```python
import numpy as np

kernel_name = "gpu_ag_gemm_m8192_k8192_n8192_bf16_silu"


ABSMAX_RTOL = 0.008

COMPUTE_DTYPE = "bfloat16"
EPILOGUE = "silu"
IN_QUANT = None
OUT_QUANT = None
IN_DTYPES = {"x": "bfloat16", "w_mat": "bfloat16"}
WORLD_SIZE = 8

def _to_bf16(x):
    u = np.ascontiguousarray(x, dtype=np.float32).view(np.uint32)
    return ((u + 0x7FFF + ((u >> 16) & 1)) & 0xFFFF0000).view(np.float32)

def setup_inputs():

    x = _to_bf16(np.random.randn(8192, 8192).astype(np.float32))
    w_mat = _to_bf16((np.random.randn(8192, 8192) / np.sqrt(8192)).astype(np.float32))
    return {"x": x, "w_mat": w_mat}

def _epilogue(y):
    return (y / (1.0 + np.exp(-np.clip(y, -60.0, 60.0)))).astype(np.float32)

def reference(x, w_mat):
    y = (x @ w_mat).astype(np.float32)
    return _epilogue(y)
```

```cpp
#include <hip/hip_runtime.h>
#include <rccl/rccl.h>
#include <cstdio>

#define HIP_CHECK(expr) do { hipError_t _e = (expr); if (_e != hipSuccess) { \
    fprintf(stderr, "HIP error %s at %s:%d\n", hipGetErrorString(_e), __FILE__, __LINE__); \
    return; } } while (0)

namespace agk {
#define LAS __attribute__((address_space(3)))
typedef unsigned u32;
typedef unsigned long long u64;
typedef unsigned u32x4 __attribute__((ext_vector_type(4)));
typedef unsigned u32x2 __attribute__((ext_vector_type(2)));
typedef short s16x4 __attribute__((ext_vector_type(4)));
typedef short s16x8 __attribute__((ext_vector_type(8)));
typedef __bf16 bf16x8 __attribute__((ext_vector_type(8)));
typedef __bf16 bf16x2 __attribute__((ext_vector_type(2)));
typedef float f32x4 __attribute__((ext_vector_type(4)));
typedef float f32x2 __attribute__((ext_vector_type(2)));

constexpr int NRANK = 8, KDIM = 8192;
constexpr int F = 12, KEPT = 32 - F;
constexpr int NWG = 256, NGEMM = 208, NCOMM = NWG - NGEMM;
constexpr int PULL_WAVES = 4;
constexpr int NCHUNK = 32;
constexpr int NSTEP = KDIM / 64;
constexpr u64 DEADLINE = 50000000ull;

constexpr size_t MiB = 1ull << 20;
constexpr size_t OFF_CTR = 0;
constexpr size_t OFF_TMO = 16384;
constexpr size_t OFF_XFLAG = 64 * 1024;
constexpr size_t OFF_WFLAG = 256 * 1024;
constexpr size_t OFF_DBG = 512 * 1024;
constexpr size_t OFF_HDR = 1 * MiB;
constexpr size_t HDR_PAR = 3 * MiB / 2;
constexpr size_t HDR_X = 0;
constexpr size_t HDR_W = 1 * MiB;
constexpr size_t HDR_O = 1 * MiB + 128 * 1024;
constexpr size_t CTRL_BYTES = 4 * MiB;
constexpr size_t OFF_XRAW = CTRL_BYTES;
constexpr size_t OFF_WRAW = OFF_XRAW + 128 * MiB;
constexpr size_t OFF_PK = OFF_WRAW + 16 * MiB;
constexpr size_t PK_PAR = 48 * MiB;
constexpr u32 PK_X = 0;
constexpr u32 PK_W = 16 * MiB;
constexpr u32 PK_O = 32 * MiB;
constexpr size_t WS_NEED = OFF_PK + 2 * PK_PAR;

constexpr int A_BYTES = 160 * 128, STAGE = A_BYTES + 64 * 512, LDS_BYTES = 3 * STAGE;
constexpr int EP_STRIDE = 528;

struct Params {
    char* ws[NRANK];
    const char* x;
    const char* w;
    char* out;
    int rank, pad;
    u64 tile[NWG];
};

#define STAMP(slot) do {} while (0)

#define WAIT_VM(n) asm volatile("s_waitcnt vmcnt(" #n ")" ::: "memory")
#define WAIT_LGKM0() do { asm volatile("" ::: "memory"); __builtin_amdgcn_s_waitcnt(0xC07F); asm volatile("" ::: "memory"); } while (0)
#define SCHED() __builtin_amdgcn_sched_barrier(0)

__device__ __forceinline__ __amdgpu_buffer_rsrc_t mkbuf(const void* p, u32 bytes) {
    return __builtin_amdgcn_make_buffer_rsrc((void*)p, 0, bytes, 0x00020000);
}
__device__ __forceinline__ u32 uni(u32 v) { return __builtin_amdgcn_readfirstlane(v); }
__device__ __forceinline__ u32 bfi(u32 m, u32 a, u32 b) { return (a & m) | (b & ~m); }

__device__ __forceinline__ bool give_up(u32* tmo, u32 call, u64 t0) {
    if (uni(__hip_atomic_load(tmo, __ATOMIC_RELAXED, __HIP_MEMORY_SCOPE_AGENT)) == call ||
        __builtin_amdgcn_s_memrealtime() - t0 > DEADLINE) {
        __hip_atomic_store(tmo, call, __ATOMIC_RELAXED, __HIP_MEMORY_SCOPE_AGENT);
        return true;
    }
    return false;
}

__device__ __forceinline__ void pack_group(const u32x4 (&r)[16], __amdgpu_buffer_rsrc_t dst, u32 slot_off,
                                           u64* hdr_ptr, int ncons, u32 call, int lane) {
    float mx = 0.f;
#pragma unroll
    for (int j = 0; j < 16; ++j) {
        asm("v_max3_f32 %0, |%1|, |%2|, %0" : "+v"(mx) : "v"(r[j][0]), "v"(r[j][1]));
        asm("v_max3_f32 %0, |%1|, |%2|, %0" : "+v"(mx) : "v"(r[j][2]), "v"(r[j][3]));
    }
    u32 top = (__float_as_uint(mx) >> 24) & 0x7fu;
#pragma unroll
    for (int o = 32; o >= 1; o >>= 1) top = max(top, (u32)__shfl_xor((int)top, o));
    top = uni(top);
    const u32 base = top > 7u ? top - 7u : 0u;
    const u32 base4 = base * 0x01010101u;
    u32 cnt = 0, hd = 0;
    u32x4 lo[8], nb[4];
#pragma unroll
    for (int j = 0; j < 16; ++j) {
        const u32 d0 = r[j][0], d1 = r[j][1], d2 = r[j][2], d3 = r[j][3];
        lo[j >> 1][(j & 1) * 2 + 0] = __builtin_amdgcn_perm(d1, d0, 0x06040200u);
        lo[j >> 1][(j & 1) * 2 + 1] = __builtin_amdgcn_perm(d3, d2, 0x06040200u);
        const u32 h0 = __builtin_amdgcn_perm(d1, d0, 0x07050301u), h1 = __builtin_amdgcn_perm(d3, d2, 0x07050301u);
        u32 c0 = h0 - base4, c1 = h1 - base4;
        if (__builtin_expect(__builtin_amdgcn_ballot_w64(((c0 | c1) & 0x78787878u) != 0u) != 0ull, 0)) {
            c0 = bfi(0x80808080u, h0, (h0 | 0x80808080u) - base4);
            c1 = bfi(0x80808080u, h1, (h1 | 0x80808080u) - base4);
            for (int e = 0; e < 8 && cnt <= 15u; ++e) {
                const u32 dw = e < 2 ? d0 : e < 4 ? d1 : e < 6 ? d2 : d3;
                const u32 v = (dw >> ((e & 1) * 16)) & 0xffffu;
                u64 m = __builtin_amdgcn_ballot_w64((((v >> 8) & 0x7fu) - base) > 7u);
                const u32 rec = ((u32)(j * 512 + lane * 8 + e) << 16) | v;
                while (m != 0ull && cnt <= 15u) {
                    const int l = __builtin_ctzll(m);
                    m &= m - 1ull;
                    ++cnt;
                    const u32 rr = __builtin_amdgcn_readlane(rec, l);
                    if ((u32)(lane & 15) == cnt) hd = rr;
                }
            }
        }
        nb[j >> 2][j & 3] = bfi(0x87878787u, c0, __builtin_amdgcn_alignbit(c1, c1, 4));
    }
    const bool raw = cnt > 15u;
    const u32 vo = slot_off + lane * 16;
    if (!raw) {
#pragma unroll
        for (int u = 0; u < 4; ++u) {
            __builtin_amdgcn_raw_buffer_store_b128(lo[2 * u], dst, vo + u * 3072, 0, 17);
            __builtin_amdgcn_raw_buffer_store_b128(lo[2 * u + 1], dst, vo + u * 3072 + 1024, 0, 17);
            __builtin_amdgcn_raw_buffer_store_b128(nb[u], dst, vo + u * 3072 + 2048, 0, 17);
        }
    } else {
        cnt = 0;
#pragma unroll
        for (int j = 0; j < 16; ++j) __builtin_amdgcn_raw_buffer_store_b128(r[j], dst, vo + j * 1024, 0, 17);
    }
    if ((lane & 15) == 0) hd = base | (cnt << 8) | (raw ? 0x10000u : 0u);
    WAIT_VM(0);
    if ((u32)(lane & 15) <= cnt && (lane >> 4) < ncons)
        __hip_atomic_store(hdr_ptr, ((u64)call << 32) | hd, __ATOMIC_RELAXED, __HIP_MEMORY_SCOPE_SYSTEM);
}

template <int AUX_ST>
__device__ __forceinline__ void pull_unit(const u64* hdr, __amdgpu_buffer_rsrc_t src, u32 slot_off, u32 unit,
                                          __amdgpu_buffer_rsrc_t dst, u32 dst_off, u32 dst_stride,
                                          u32* tmo, u32 call, int lane, bool nan_on_timeout) {
    u32 hd, meta;
    bool dead = false;
    {
        const u64 t0 = __builtin_amdgcn_s_memrealtime();
        for (;;) {
            const u64 g = __hip_atomic_load(hdr + (lane & 15), __ATOMIC_RELAXED, __HIP_MEMORY_SCOPE_SYSTEM);
            hd = (u32)g;
            meta = uni(hd);
            const u64 ok = __builtin_amdgcn_ballot_w64((u32)(g >> 32) == call);
            const u64 need = (2ull << ((meta >> 8) & 15u)) - 1ull;
            if ((ok & need) == need) break;
            if (give_up(tmo, call, t0)) { dead = true; break; }
            __builtin_amdgcn_s_sleep(2);
        }
    }
    asm volatile("" ::: "memory");
    u32x4 o[4];
    if (!(meta & 0x10000u)) {
        const u32 vo = slot_off + unit * 3072 + lane * 16;
        const u32x4 la = __builtin_amdgcn_raw_buffer_load_b128(src, vo, 0, 17);
        const u32x4 lb = __builtin_amdgcn_raw_buffer_load_b128(src, vo + 1024, 0, 17);
        const u32x4 nb = __builtin_amdgcn_raw_buffer_load_b128(src, vo + 2048, 0, 17);
        const u32 base4 = (meta & 0x7fu) * 0x01010101u;
#pragma unroll
        for (int j = 0; j < 4; ++j) {
            const u32 h0 = (nb[j] & 0x87878787u) + base4;
            const u32 h1 = (__builtin_amdgcn_alignbit(nb[j], nb[j], 28) & 0x87878787u) + base4;
            const u32 l0 = j < 2 ? la[(j & 1) * 2] : lb[(j & 1) * 2], l1 = j < 2 ? la[(j & 1) * 2 + 1] : lb[(j & 1) * 2 + 1];
            o[j][0] = __builtin_amdgcn_perm(h0, l0, 0x05010400u);
            o[j][1] = __builtin_amdgcn_perm(h0, l0, 0x07030602u);
            o[j][2] = __builtin_amdgcn_perm(h1, l1, 0x05010400u);
            o[j][3] = __builtin_amdgcn_perm(h1, l1, 0x07030602u);
        }
        const u32 cnt = (meta >> 8) & 15u;
        for (u32 i = 1; i <= cnt; ++i) {
            const u32 rec = __builtin_amdgcn_readlane(hd, i);
            const u32 pos = rec >> 16;
            if ((pos >> 11) != unit) continue;
            const u32 dsel = ((pos >> 9) & 3u) * 4u + ((pos >> 1) & 3u);
            const u32 m = ((pos >> 3) & 63u) == (u32)lane ? ((pos & 1u) ? 0xffff0000u : 0x0000ffffu) : 0u;
            const u32 val = (rec & 0xffffu) * 0x10001u;
#pragma unroll
            for (int d = 0; d < 16; ++d) o[d >> 2][d & 3] = bfi(dsel == (u32)d ? m : 0u, val, o[d >> 2][d & 3]);
        }
    } else {
        const u32 vo = slot_off + unit * 4096 + lane * 16;
#pragma unroll
        for (int j = 0; j < 4; ++j) o[j] = __builtin_amdgcn_raw_buffer_load_b128(src, vo + j * 1024, 0, 17);
    }
    if (nan_on_timeout) {
        if (dead || uni(__hip_atomic_load(tmo, __ATOMIC_RELAXED, __HIP_MEMORY_SCOPE_AGENT)) == call) {
#pragma unroll
            for (int j = 0; j < 4; ++j) o[j] = (u32x4){0x7fc07fc0u, 0x7fc07fc0u, 0x7fc07fc0u, 0x7fc07fc0u};
        }
    }
    const u32 dv = dst_off + (lane >> 5) * dst_stride + (lane & 31) * 16;
#pragma unroll
    for (int j = 0; j < 4; ++j) __builtin_amdgcn_raw_buffer_store_b128(o[j], dst, dv + 2 * j * dst_stride, 0, AUX_ST);
}

__device__ __forceinline__ void glds16(const char* sbase, u32 voff, u32 lds_addr) {
    asm volatile("s_mov_b32 m0, %2\n\ts_nop 0\n\tglobal_load_lds_dwordx4 %0, %1 sc0 sc1"
                 :: "v"(voff), "s"(sbase), "s"(lds_addr) : "memory");
}
__device__ __forceinline__ void glds16_half(const char* sbase, u32 voff, u32 lds_addr) {
    asm volatile("s_mov_b32 m0, %2\n\ts_mov_b32 exec_hi, 0\n\tglobal_load_lds_dwordx4 %0, %1 sc0 sc1\n\ts_mov_b32 exec_hi, -1"
                 :: "v"(voff), "s"(sbase), "s"(lds_addr) : "memory");
}

__device__ __forceinline__ float silu(float y) {
    const float c = __builtin_amdgcn_fmed3f(y, -60.f, 60.f);
    return y * __builtin_amdgcn_rcpf(1.f + __builtin_amdgcn_exp2f(c * -1.44269504088896f));
}

__global__ void __launch_bounds__(512, 2) ag_gemm_silu(const Params P) {
    __shared__ __attribute__((aligned(16))) char lds_raw[LDS_BYTES];
    LAS char* lds = (LAS char*)lds_raw;
    const int lane = threadIdx.x & 63, wid = uni(threadIdx.x >> 6);
    const int rank = P.rank, grp = rank & 4, oth = grp ^ 4;
    char* const ws = P.ws[rank];
    const u64 tile = P.tile[blockIdx.x];
    const u32 cb = (u32)tile & 0xffu, ridx = (u32)(tile >> 56);
    u32* const tmo = (u32*)(ws + OFF_TMO);
    STAMP(0);
    u32* const ctr = (u32*)(ws + OFF_CTR) + blockIdx.x * 8 + wid;
    const u32 call = uni(__hip_atomic_load(ctr, __ATOMIC_RELAXED, __HIP_MEMORY_SCOPE_AGENT)) + 1u;
    if (lane == 0) __hip_atomic_store(ctr, call, __ATOMIC_RELAXED, __HIP_MEMORY_SCOPE_AGENT);
    const u32 par = call & 1u;
    const size_t hdr_par = OFF_HDR + par * HDR_PAR;

    if (cb == 0xffu) {
        if (wid >= PULL_WAVES) return;
        constexpr int PER = NCOMM * PULL_WAVES / 7;
        const u32 gw = ridx * PULL_WAVES + wid;
        if (gw >= PER * 7) return;
        const u32 dir = gw % 7 + 1, j0 = gw / 7;
        const u32 p = (rank ^ dir) & 7u;
        const auto src = mkbuf(P.ws[p] + OFF_PK + par * PK_PAR, PK_PAR);
        const auto xraw = mkbuf(ws + OFF_XRAW, 128 * MiB);
        const char* hdr = ws + hdr_par;
        u32* const xflag = (u32*)(ws + OFF_XFLAG);
        if (dir < 4) {
            for (u32 n = j0; n < NCHUNK * 4 * KEPT; n += PER) {
                const u32 c = n / (4 * KEPT), rem = n % (4 * KEPT), t = rem >> 2, u = rem & 3, gs = p * 32 + t;
                pull_unit<17>((const u64*)(hdr + HDR_X + (gs * 32 + c) * 128), src, PK_X + (c * 32 + t) * 16384, u,
                              xraw, (gs * 32 + u * 8) * 16384 + c * 512, 16384, tmo, call, lane, false);
                WAIT_VM(0);
                if (lane == 0) __hip_atomic_store(xflag + (gs * 32 + c) * 4 + u, call, __ATOMIC_RELAXED, __HIP_MEMORY_SCOPE_SYSTEM);
            }
            STAMP(1);
        } else {
            const auto wraw = mkbuf(ws + OFF_WRAW, 16 * MiB);
            u32* const wflag = (u32*)(ws + OFF_WFLAG);
            constexpr u32 PC = 4 * F + 32;
            u32 n = j0;
            for (; n < NCHUNK * PC; n += PER) {
                const u32 c = n / PC, rem = n % PC;
                if (rem < 32) {
                    const u32 kg = c * 8 + (rem >> 2), u = rem & 3;
                    pull_unit<17>((const u64*)(hdr + HDR_W + ((p & 3) * 256 + kg) * 128), src, PK_W + (kg * 4 + (rank & 3)) * 16384, u,
                                  wraw, (kg * 32 + u * 8) * 2048 + (p & 3) * 512, 2048, tmo, call, lane, false);
                    WAIT_VM(0);
                    if (lane == 0) __hip_atomic_store(wflag + ((p & 3) * 32 + c) * 32 + rem, call, __ATOMIC_RELAXED, __HIP_MEMORY_SCOPE_SYSTEM);
                } else {
                    const u32 t = KEPT + ((rem - 32) >> 2), u = rem & 3, gs = p * 32 + t;
                    pull_unit<17>((const u64*)(hdr + HDR_X + (gs * 32 + c) * 128), src, PK_X + (c * 32 + t) * 16384, u,
                                  xraw, (gs * 32 + u * 8) * 16384 + c * 512, 16384, tmo, call, lane, false);
                    WAIT_VM(0);
                    if (lane == 0) __hip_atomic_store(xflag + (gs * 32 + c) * 4 + u, call, __ATOMIC_RELAXED, __HIP_MEMORY_SCOPE_SYSTEM);
                }
            }
            STAMP(1);
            const auto out = mkbuf(P.out, 16 * MiB);
            for (n -= NCHUNK * PC; n < (128 - F) * 4; n += PER) {
                const u32 i = n >> 2, u = n & 3;
                u32 gs;
                if (i < 4 * KEPT) gs = ((p ^ (i / KEPT)) & 7u) * 32 + i % KEPT;
                else { const u32 q = i - 4 * KEPT; gs = ((rank ^ (1 + q / F)) & 7u) * 32 + KEPT + q % F; }
                pull_unit<0>((const u64*)(hdr + HDR_O + ((p & 3) * 256 + gs) * 128), src, PK_O + ((rank & 3) * 256 + gs) * 16384, u,
                             out, (gs * 32 + u * 8) * 2048 + (p & 3) * 512, 2048, tmo, call, lane, true);
            }
            STAMP(2);
        }
        return;
    }

    const auto pk = mkbuf(ws + OFF_PK + par * PK_PAR, PK_PAR);
    for (u32 J = wid * NGEMM + ridx; J < 2048; J += 8 * NGEMM) {
        const u32 c = J >> 6, r6 = J & 63;
        if (J < 8 * NGEMM) { for (u32 i = 0; i < c; ++i) __builtin_amdgcn_s_sleep(12); }
        else __builtin_amdgcn_s_sleep(127);
        u32x4 r[16];
        u32 slot;
        u64* hp;
        int ncons;
        if (r6 < 32) {
            const auto xs = mkbuf(P.x, 16 * MiB);
            const u32 vo = (r6 * 32 + (lane >> 5)) * 16384 + c * 512 + (lane & 31) * 16;
#pragma unroll
            for (int j = 0; j < 16; ++j) r[j] = __builtin_amdgcn_raw_buffer_load_b128(xs, vo + j * 32768, 0, 0);
            slot = PK_X + (c * 32 + r6) * 16384;
            const bool kept = r6 < KEPT;
            ncons = kept ? 3 : 4;
            char* b = nullptr;
#pragma unroll
            for (int q = 0; q < 4; ++q) { char* pq = P.ws[(rank ^ (kept ? (q < 3 ? q + 1 : 1) : 4 + q)) & 7]; if ((lane >> 4) == q) b = pq; }
            hp = (u64*)(b + hdr_par + HDR_X + ((rank * 32 + r6) * 32 + c) * 128) + (lane & 15);
        } else {
            const auto wsrc = mkbuf(P.w, 16 * MiB);
            const u32 q = r6 & 3, kg = c * 8 + ((r6 - 32) >> 2);
            const u32 vo = (kg * 32 + (lane >> 5)) * 2048 + q * 512 + (lane & 31) * 16;
#pragma unroll
            for (int j = 0; j < 16; ++j) r[j] = __builtin_amdgcn_raw_buffer_load_b128(wsrc, vo + j * 4096, 0, 0);
            slot = PK_W + (kg * 4 + q) * 16384;
            ncons = 1;
            hp = (u64*)(P.ws[(oth | q) & 7] + hdr_par + HDR_W + ((rank & 3) * 256 + kg) * 128) + (lane & 15);
        }
        pack_group(r, pk, slot, hp, ncons, call, lane);
    }
    STAMP(1);

    const int wm = wid >> 2, wn = wid & 3, fr = lane & 15, fq = lane >> 4;
    const u32 ns = (u32)(tile >> 8) & 0xffu;
    auto strip_of = [&](u32 i) -> u32 { return (u32)(tile >> (16 + 8 * i)) & 0xffu; };
    auto strip_base = [&](u32 gs) -> const char* {
        return ((gs >> 5) == (u32)rank) ? P.x + (size_t)(gs & 31) * (32 * 16384) : ws + OFF_XRAW + (size_t)gs * (32 * 16384);
    };
    const char* gA01 = strip_base(strip_of(wid >> 1));
    const char* gA2 = strip_base(strip_of(4));
    const char* gB = (cb < 4 ? P.w : ws + OFF_WRAW) + (cb & 3) * 512;
    const u32 l3 = lane >> 3;
    const u32 vA0 = ((wid & 1) * 16 + l3) * 16384 + (((lane & 7) ^ (l3 >> 1)) * 16);
    const u32 vA1 = ((wid & 1) * 16 + 8 + l3) * 16384 + (((lane & 7) ^ (4 + (l3 >> 1))) * 16);
    const u32 vA2 = (4 * wid + (l3 & 3)) * 16384 + (((lane & 7) ^ (((4 * wid + (l3 & 3)) >> 1) & 7)) * 16);
    u32 vB[4];
#pragma unroll
    for (int i = 0; i < 4; ++i) {
        const u32 k = 8 * wid + 2 * i + (lane >> 5), f = (k & 3) | (((k >> 3) & 1) << 2);
        vB[i] = k * 2048 + (((lane & 31) ^ (f << 1)) * 16);
    }
    const u32 lds0 = (u32)(size_t)lds;
    const u32 rA0 = (wm * 80 + fr) * 128 + ((fq ^ (fr >> 1)) * 16), rA1 = rA0 ^ 64;
    u32 rB[4];
    {
        const u32 q = fr >> 2, p4 = fr & 3, f = q | ((fq & 1) << 2);
#pragma unroll
        for (int n = 0; n < 4; ++n) rB[n] = A_BYTES + (8 * fq + q) * 512 + (((wn * 4 + n) ^ f) * 32) + p4 * 8;
    }

    u32 ready = 0;
    u32 poll_off; bool poll_on;
    {
        const u32 i = lane & 15, jj = lane >> 4;
        const u32 gs = strip_of(i < 5 ? i : 0);
        if (i < 5) { poll_on = (gs >> 5) != (u32)rank; poll_off = OFF_XFLAG + (gs * 32 + jj) * 16; }
        else { poll_on = cb >= 4 && i < 13; poll_off = OFF_WFLAG + ((cb & 3) * 32 + jj) * 128 + ((i - 5) & 7) * 16; }
    }
    const u32 poll_mul = (lane & 15) < 5 ? 16 : 128;
    const auto ctrl = mkbuf(ws, CTRL_BYTES);
    auto gate = [&](u32 step) {
        if (wid == 0 && step >= ready) {
            const u32 c = step >> 2;
            const u64 t0 = __builtin_amdgcn_s_memrealtime();
            for (;;) {
                const u32x4 v = __builtin_amdgcn_raw_buffer_load_b128(ctrl, poll_off + c * poll_mul, 0, 17);
                const bool ok = !poll_on || (v[0] == call && v[1] == call && v[2] == call && v[3] == call);
                const u64 m = ~__builtin_amdgcn_ballot_w64(ok);
                const u32 n = m == 0ull ? 4u : (u32)__builtin_ctzll(m) >> 4;
                if (n != 0u) { ready = (c + n) * 4; break; }
                if (give_up(tmo, call, t0)) { ready = NSTEP + 16; break; }
                __builtin_amdgcn_s_sleep(4);
            }
            asm volatile("" ::: "memory");
        }
    };
    auto stage_piece = [&](int i, u32 slot_base) {
        if (i == 0) glds16(gA01, vA0, slot_base + wid * 2048);
        else if (i == 1) glds16(gA01, vA1, slot_base + wid * 2048 + 1024);
        else if (i == 2) glds16_half(gA2, vA2, slot_base + 16384 + wid * 512);
        else glds16(gB, vB[i - 3], slot_base + A_BYTES + wid * 4096 + (i - 3) * 1024);
    };
    auto advance = [&]() { gA01 += 128; gA2 += 128; gB += 64 * 2048; };

    f32x4 acc[5][4];
#pragma unroll
    for (int m = 0; m < 5; ++m)
#pragma unroll
        for (int n = 0; n < 4; ++n) acc[m][n] = (f32x4){0.f, 0.f, 0.f, 0.f};
    s16x8 fa[2][5];
    s16x4 fb[2][4][2];
    auto frag_read = [&](int set, int i, u32 sb, int kk) {
        if (i == 2) fa[set][0] = *(const LAS s16x8*)(lds + sb + (kk ? rA1 : rA0));
        else if (i >= 9) fa[set][i - 8] = *(const LAS s16x8*)(lds + sb + (kk ? rA1 : rA0) + (i - 8) * 2048);
        else {
            const int b = i < 2 ? i : i - 1, n = b >> 1, t = b & 1;
            fb[set][n][t] = __builtin_amdgcn_ds_read_tr16_b64_v4i16((LAS s16x4*)(lds + sb + rB[n] + t * 2048 + kk * 16384));
        }
    };
    auto mma = [&](int set, int i) {
        const int m = i >> 2, n = i & 3;
        const s16x8 b = __builtin_shufflevector(fb[set][n][0], fb[set][n][1], 0, 1, 2, 3, 4, 5, 6, 7);
        acc[m][n] = __builtin_amdgcn_mfma_f32_16x16x32_bf16(__builtin_bit_cast(bf16x8, b), __builtin_bit_cast(bf16x8, fa[set][m]), acc[m][n], 0, 0, 0);
    };

    gate(0);
    __builtin_amdgcn_s_barrier();
    STAMP(2);
#pragma unroll
    for (int s = 0; s < 3; ++s) {
#pragma unroll
        for (int i = 0; i < 7; ++i) stage_piece(i, lds0 + s * STAGE);
        advance();
    }
    WAIT_VM(14);
    __builtin_amdgcn_s_barrier();
    asm volatile("" ::: "memory");
#pragma unroll
    for (int i = 0; i < 13; ++i) frag_read(0, i, 0, 0);
    u32 cur = 0;
#define KSTEP(LOAD, VMW, NEXT) do { \
        const u32 nxt = cur == 2 * STAGE ? 0u : cur + STAGE; \
        _Pragma("unroll") for (int i = 0; i < 20; ++i) { mma(0, i); SCHED(); if (i < 13) frag_read(1, i, cur, 1); SCHED(); } \
        if (LOAD) gate(t + 3); \
        WAIT_LGKM0(); \
        if (NEXT) { WAIT_VM(VMW); __builtin_amdgcn_s_barrier(); asm volatile("" ::: "memory"); } \
        SCHED(); \
        _Pragma("unroll") for (int i = 0; i < 20; ++i) { \
            mma(1, i); SCHED(); \
            if (NEXT && i < 13) frag_read(0, i, nxt, 0); \
            if (LOAD && i >= 13) stage_piece(i - 13, lds0 + cur); \
            SCHED(); } \
        if (LOAD) advance(); \
        cur = nxt; } while (0)
    for (u32 t = 0; t < NSTEP - 3; ++t) KSTEP(true, 7, true);
    { const u32 t = 0; (void)t; KSTEP(false, 7, true); KSTEP(false, 0, true); KSTEP(false, 0, false); }
#undef KSTEP
    STAMP(3);

    const bool bad = uni(__hip_atomic_load(tmo, __ATOMIC_RELAXED, __HIP_MEMORY_SCOPE_AGENT)) == call;
    __builtin_amdgcn_s_barrier();
    asm volatile("" ::: "memory");
#pragma unroll
    for (int m = 0; m < 5; ++m)
#pragma unroll
        for (int n = 0; n < 4; ++n) {
            const f32x4 a = acc[m][n];
            u32x2 o;
            o[0] = __builtin_bit_cast(u32, __builtin_convertvector((f32x2){silu(a[0]), silu(a[1])}, bf16x2));
            o[1] = __builtin_bit_cast(u32, __builtin_convertvector((f32x2){silu(a[2]), silu(a[3])}, bf16x2));
            if (bad) o = (u32x2){0x7fc07fc0u, 0x7fc07fc0u};
            *(LAS u32x2*)(lds + (wm * 80 + m * 16 + fr) * EP_STRIDE + (wn * 64 + n * 16 + fq * 4) * 2) = o;
        }
    WAIT_LGKM0();
    __builtin_amdgcn_s_barrier();
    asm volatile("" ::: "memory");
    if (cb < 4) {
        const auto out = mkbuf(P.out, 16 * MiB);
#pragma unroll
        for (int i = 0; i < 10; ++i) {
            const u32 pc = i * 8 + wid, s = pc >> 4;
            if (s < ns) {
                const u32 row = 2 * pc + (lane >> 5);
                const u32x4 v = *(const LAS u32x4*)(lds + row * EP_STRIDE + (lane & 31) * 16);
                __builtin_amdgcn_raw_buffer_store_b128(v, out, (strip_of(s) * 32 + (row & 31)) * 2048 + cb * 512 + (lane & 31) * 16, 0, 0);
            }
        }
    } else if ((u32)wid < ns) {
        u32x4 r[16];
#pragma unroll
        for (int j = 0; j < 16; ++j) r[j] = *(const LAS u32x4*)(lds + (wid * 32 + 2 * j + (lane >> 5)) * EP_STRIDE + (lane & 31) * 16);
        const u32 gs = strip_of(wid), q = cb & 3;
        u64* hp = (u64*)(P.ws[(oth | q) & 7] + hdr_par + HDR_O + ((rank & 3) * 256 + gs) * 128) + (lane & 15);
        pack_group(r, pk, PK_O + (q * 256 + gs) * 16384, hp, 1, call, lane);
    }
    STAMP(4);
}

static void build_tiles(int rank, u64* tile) {
    const int grp = rank & 4, oth = grp ^ 4;
    int nl[8] = {0};
    u64 list[8][32];
    for (int cb = 0; cb < 8; ++cb) {
        int q[4][64], nq[4] = {0, 0, 0, 0};
        for (int i = 0; i < 4; ++i) {
            for (int t = 0; t < KEPT; ++t) q[i][nq[i]++] = (grp | i) * 32 + t;
            if (cb < 4 || (cb & 3) != i)
                for (int t = KEPT; t < 32; ++t) q[i][nq[i]++] = (oth | i) * 32 + t;
        }
        if (cb < 4) for (int t = KEPT; t < 32; ++t) { const int i = (t - KEPT) & 3; q[i][nq[i]++] = rank * 32 + t; }
        const int cap = cb < 4 ? 35 : 30;
        for (int i = 0; i < 4; ++i)
            while (nq[i] > cap) {
                int d = 0;
                for (int k = 1; k < 4; ++k) if (nq[k] < nq[d]) d = k;
                q[d][nq[d]++] = q[i][--nq[i]];
            }
        for (int i = 0; i < 4; ++i) {
            const int xcd = i + 4 * ((cb >> 1) & 1);
            for (int s0 = 0; s0 < cap; s0 += 5) {
                int n = nq[i] - s0; n = n > 5 ? 5 : n;
                u64 e = (u64)cb;
                if (n <= 0) { n = 0; for (int s = 0; s < 5; ++s) e |= (u64)(rank * 32) << (16 + 8 * s); }
                else for (int s = 0; s < 5; ++s) e |= (u64)q[i][s0 + (s < n ? s : 0)] << (16 + 8 * s);
                e |= (u64)n << 8;
                list[xcd][nl[xcd]++] = e;
            }
        }
    }
    int ng = 0, nc = 0;
    for (int b = 0; b < NWG; ++b) {
        const int xcd = b & 7, i = b >> 3;
        if (i < NCOMM / 8) tile[b] = 0xffull | ((u64)nc++ << 56);
        else tile[b] = list[xcd][i - NCOMM / 8] | ((u64)ng++ << 56);
    }
}
}

extern "C" void kernel_launch(int rank, int nranks, ncclComm_t comm,
                              hipStream_t stream,
                              void* const* d_in, const int* in_sizes, int n_in,
                              void* d_out, int out_size,
                              const int* in_dtypes, int out_dtype,
                              void* const* d_out_all,
                              void* const* d_ws_all, size_t ws_bytes) {
    using namespace agk;
    if (nranks != NRANK || (unsigned)rank >= (unsigned)NRANK || ws_bytes < WS_NEED || n_in < 2 ||
        in_sizes[0] != 1024 * KDIM || in_sizes[1] != KDIM * 1024 || out_size != 8192 * 1024) {
        fprintf(stderr, "kernel_launch: unexpected problem shape\n");
        return;
    }
    Params P;
    for (int p = 0; p < NRANK; ++p) P.ws[p] = (char*)d_ws_all[p];
    P.x = (const char*)d_in[0];
    P.w = (const char*)d_in[1];
    P.out = (char*)d_out;
    P.rank = rank;
    P.pad = 0;
    build_tiles(rank, P.tile);
    hipLaunchKernelGGL(ag_gemm_silu, dim3(NWG), dim3(512), 0, stream, P);
    HIP_CHECK(hipGetLastError());
}
```

```cpp
#include <hip/hip_runtime.h>
#include <rccl/rccl.h>
#include <cstdio>

#define HIP_CHECK(expr) do { hipError_t _e = (expr); if (_e != hipSuccess) { \
    fprintf(stderr, "HIP error %s at %s:%d\n", hipGetErrorString(_e), __FILE__, __LINE__); \
    return; } } while (0)

namespace agk {
#define LAS __attribute__((address_space(3)))
typedef unsigned u32;
typedef unsigned long long u64;
typedef unsigned u32x4 __attribute__((ext_vector_type(4)));
typedef unsigned u32x2 __attribute__((ext_vector_type(2)));
typedef short s16x4 __attribute__((ext_vector_type(4)));
typedef short s16x8 __attribute__((ext_vector_type(8)));
typedef __bf16 bf16x8 __attribute__((ext_vector_type(8)));
typedef __bf16 bf16x2 __attribute__((ext_vector_type(2)));
typedef float f32x4 __attribute__((ext_vector_type(4)));
typedef float f32x2 __attribute__((ext_vector_type(2)));

constexpr int NRANK = 8, KDIM = 8192;
constexpr int F = 12, KEPT = 32 - F;
constexpr int NWG = 256, NGEMM = 208, NCOMM = NWG - NGEMM;
#define AGK_PER 27
#define AGK_MS 6
#define AGK_MO 10
#define AGK_XLAST 1
#define AGK_HS 16
#define AGK_PSLEEP 2
#define AGK_KA 10
#define AGK_LI 12
#define AGK_LC 20
constexpr int PER = AGK_PER;
constexpr int KA = AGK_KA;
constexpr int LEAD_I = AGK_LI, LEAD_C = AGK_LC;
constexpr int MIX_STEPS = AGK_MS, MIX_OUT = AGK_MO;
constexpr int NCHUNK = 32;
constexpr int NSTEP = KDIM / 64;
constexpr u32 HEAD_STEPS = AGK_HS, HEAD_JOBS = HEAD_STEPS * (KA + 32), NJOBS = HEAD_JOBS + 2048 - HEAD_STEPS / 4 * (KA + 32);
constexpr int NSPARE = NCOMM * 9 - PER * 7;
static_assert(PER * 7 <= NCOMM * 8, "spare comm waves needed");
constexpr u64 DEADLINE = 50000000ull;

constexpr size_t MiB = 1ull << 20;
constexpr size_t OFF_CTR = 0;
constexpr size_t OFF_TMO = 16384;
constexpr size_t OFF_TICKET = 32 * 1024;
constexpr size_t OFF_XFLAG = 64 * 1024;
constexpr size_t OFF_WFLAG = 256 * 1024;
constexpr size_t OFF_DBG = 512 * 1024;
constexpr size_t OFF_HDR = 1 * MiB;
constexpr size_t HDR_PAR = MiB / 2;
constexpr size_t HDR_X = 0;
constexpr size_t HDR_W = 256 * 1024;
constexpr size_t HDR_O = 320 * 1024;
constexpr size_t CTRL_BYTES = 4 * MiB;
constexpr size_t OFF_XRAW = CTRL_BYTES;
constexpr size_t OFF_WRAW = OFF_XRAW + 128 * MiB;
constexpr size_t OFF_PK = OFF_WRAW + 16 * MiB;
constexpr size_t PK_PAR = 48 * MiB;
constexpr u32 PK_X = 0;
constexpr u32 PK_W = 16 * MiB;
constexpr u32 PK_O = 32 * MiB;
constexpr size_t WS_NEED = OFF_PK + 2 * PK_PAR;

constexpr int A_BYTES = 160 * 128, STAGE = A_BYTES + 64 * 512, LDS_BYTES = 3 * STAGE;
constexpr int EP_STRIDE = 528;
constexpr int LDS_CODEC = 96 * 1024, CODEC_BYTES = 3072;

struct Params {
    char* ws[NRANK];
    const char* x;
    const char* w;
    char* out;
    char* self;
    int rank, n_early, n_remote, pad;
    u64 tile[NWG];
};

#define STAMP(slot) do {} while (0)

#define WAIT_VM(n) asm volatile("s_waitcnt vmcnt(" #n ")" ::: "memory")
#define WAIT_LGKM0() do { asm volatile("" ::: "memory"); __builtin_amdgcn_s_waitcnt(0xC07F); asm volatile("" ::: "memory"); } while (0)
#define SCHED() __builtin_amdgcn_sched_barrier(0)

__device__ __forceinline__ __amdgpu_buffer_rsrc_t mkbuf(const void* p, u32 bytes) {
    return __builtin_amdgcn_make_buffer_rsrc((void*)p, 0, bytes, 0x00020000);
}
__device__ __forceinline__ u32 uni(u32 v) { return __builtin_amdgcn_readfirstlane(v); }
__device__ __forceinline__ u32 bfi(u32 m, u32 a, u32 b) { return (a & m) | (b & ~m); }

__device__ __forceinline__ bool give_up(u32* tmo, u32 call, u64 t0) {
    if (uni(__hip_atomic_load(tmo, __ATOMIC_RELAXED, __HIP_MEMORY_SCOPE_AGENT)) == call ||
        __builtin_amdgcn_s_memrealtime() - t0 > DEADLINE) {
        __hip_atomic_store(tmo, call, __ATOMIC_RELAXED, __HIP_MEMORY_SCOPE_AGENT);
        return true;
    }
    return false;
}

typedef unsigned short u16x2 __attribute__((ext_vector_type(2)));
typedef unsigned u32x3 __attribute__((ext_vector_type(3)));
__device__ __forceinline__ u32 pk_add(u32 a, u32 b) { return __builtin_bit_cast(u32, __builtin_bit_cast(u16x2, a) + __builtin_bit_cast(u16x2, b)); }
__device__ __forceinline__ u32 pk_sub(u32 a, u32 b) { return __builtin_bit_cast(u32, __builtin_bit_cast(u16x2, a) - __builtin_bit_cast(u16x2, b)); }
__device__ __forceinline__ u32 pk_min(u32 a, u32 b) { return __builtin_bit_cast(u32, __builtin_elementwise_min(__builtin_bit_cast(u16x2, a), __builtin_bit_cast(u16x2, b))); }
__device__ __forceinline__ u32 perm(u32 hi, u32 lo, u32 sel) { return __builtin_amdgcn_perm(hi, lo, sel); }
__device__ __forceinline__ u32 wave_scan(u32 v) {
    v += (u32)__builtin_amdgcn_update_dpp(0, (int)v, 0x111, 0xf, 0xf, true);
    v += (u32)__builtin_amdgcn_update_dpp(0, (int)v, 0x112, 0xf, 0xf, true);
    v += (u32)__builtin_amdgcn_update_dpp(0, (int)v, 0x114, 0xf, 0xf, true);
    v += (u32)__builtin_amdgcn_update_dpp(0, (int)v, 0x118, 0xf, 0xf, true);
    v += (u32)__builtin_amdgcn_update_dpp(0, (int)v, 0x142, 0xa, 0xf, false);
    v += (u32)__builtin_amdgcn_update_dpp(0, (int)v, 0x143, 0xc, 0xf, false);
    return v;
}
__device__ __forceinline__ u32 sym_to_d(u32 s) { return perm(0xfffefdfcu, 0xfbfa0403u, s); }
__device__ __forceinline__ u32 compact_table(int lane) {
    u32 t = 0x0c0c0c0cu;
#pragma unroll
    for (int b = 3; b >= 0; --b) if ((lane >> b) & 1) t = (t << 8) | (u32)b;
    return t;
}
__device__ __forceinline__ u32 choose_window(u32x4 v) {
    u32 sum = 0;
#pragma unroll
    for (int i = 0; i < 4; ++i) sum = __builtin_amdgcn_udot2(__builtin_bit_cast(u16x2, v[i] & 0x7fff7fffu), (u16x2){1, 1}, sum, false);
    const u32 mean = (u32)__builtin_amdgcn_readlane(wave_scan(sum), 63) >> 9;
    return mean > 133u ? (mean - 133u) >> 3 : 0u;
}

template <class Pre>
__device__ __forceinline__ u32 pack_unit(const u32x4 (&r)[4], u32 win, u32 ctab, LAS char* st,
                                         __amdgpu_buffer_rsrc_t dst, u32 slot, int lane, Pre before_stores) {
    const u32 B2 = ((win & 0xfffu) << 4) * 0x10001u;
    u32 L[8], D[8], C[2] = {0u, 0u};
#pragma unroll
    for (int k = 0; k < 8; ++k) {
        const u32 d0 = r[k >> 1][(k & 1) * 2], d1 = r[k >> 1][(k & 1) * 2 + 1];
        const u32 x0 = pk_sub(__builtin_amdgcn_alignbit(d0, d0, 31), B2), x1 = pk_sub(__builtin_amdgcn_alignbit(d1, d1, 31), B2);
        L[k] = perm(x1, x0, 0x06040200u);
        D[k] = perm(x1, x0, 0x07050301u);
        C[k >> 2] |= perm(pk_min(x1, 0x03ff03ffu), pk_min(x0, 0x03ff03ffu), 0x07050301u) << (2 * (k & 3));
    }
    const u32 E[2] = {C[0] & (C[0] >> 1) & 0x55555555u, C[1] & (C[1] >> 1) & 0x55555555u};
    *(LAS u32x4*)(st + lane * 16) = (u32x4){0u, 0u, 0u, 0u};
    *(LAS u32x2*)(st + 1024 + lane * 8) = (u32x2){0u, 0u};
    u32 f[8], sel[8];
#pragma unroll
    for (int k = 0; k < 8; ++k) {
        f[k] = (E[k >> 2] >> (2 * (k & 3))) & 0x01010101u;
        sel[k] = (u32)__builtin_amdgcn_ds_bpermute((int)__builtin_amdgcn_udot4(f[k], 0x20100804u, 0u, false), (int)ctab);
    }
    SCHED();
    const u32 tot = __builtin_popcount(E[0]) + __builtin_popcount(E[1]);
    const u32 incl = wave_scan(tot), n = __builtin_amdgcn_readlane(incl, 63);
#pragma unroll
    for (int k = 0; k < 8; ++k) { D[k] = perm(0u, D[k], sel[k]); asm volatile("" : "+v"(D[k])); }
    const u32 nsw = (n + 9u) / 10u;
    u32 np = 0, t0 = 0, t1 = 0;
    bool raw = nsw > 128u;
    if (!raw) {
        asm volatile("" ::: "memory");
        u32 p = incl - tot + (u32)(size_t)st;
#pragma unroll
        for (int k = 0; k < 8; ++k) {
            const u64 v = (u64)D[k] << ((p & 3u) * 8u);
            LAS u32* q = (LAS u32*)(size_t)(p & ~3u);
            __hip_atomic_fetch_or(q, (u32)v, __ATOMIC_RELAXED, __HIP_MEMORY_SCOPE_WAVEFRONT);
            __hip_atomic_fetch_or(q + 1, (u32)(v >> 32), __ATOMIC_RELAXED, __HIP_MEMORY_SCOPE_WAVEFRONT);
            p += __builtin_popcount(f[k]);
        }
        asm volatile("" ::: "memory");
        *(LAS unsigned char*)(st + n + (lane & 15)) = 0xff;
        asm volatile("" ::: "memory");
        LAS u32* const list = (LAS u32*)(st + 2048);
        for (u32 i0 = 0; i0 < nsw; i0 += 64) {
            const u32 i = i0 + lane;
            const LAS u32* q = (const LAS u32*)(st + ((i * 10u) & ~3u));
            const u32 a0 = q[0], a1 = q[1], a2 = q[2], sh = (lane & 1) * 2;
            const u32 d[3] = {__builtin_amdgcn_alignbyte(a1, a0, sh), __builtin_amdgcn_alignbyte(a2, a1, sh), __builtin_amdgcn_alignbyte(0u, a2, sh)};
            u32 sy[3], x[3];
#pragma unroll
            for (int k = 0; k < 3; ++k) {
                const u32 pos = (~d[k] >> 7) & 0x01010101u;
                sy[k] = ((d[k] & 0x07070707u) + pos * 5u) & 0x07070707u;
                x[k] = sym_to_d(sy[k]) ^ d[k];
            }
            x[2] &= 0xffffu;
            if (i >= nsw) x[0] = x[1] = x[2] = 0u;
            const u32 w = sy[0] | (sy[1] << 3) | (((sy[2] & 0x0303u) | ((sy[2] & 0x0404u) << 14)) << 6);
            if (i0 == 0) t0 = w; else t1 = w;
            for (;;) {
                const bool has = (x[0] | x[1] | x[2]) != 0u;
                const u64 m = __builtin_amdgcn_ballot_w64(has);
                if (m == 0ull || np > 127u) break;
                const u32 ws = x[0] ? 0u : x[1] ? 1u : 2u;
                const u32 xw = x[0] ? x[0] : x[1] ? x[1] : x[2], dw = x[0] ? d[0] : x[1] ? d[1] : d[2];
                const u32 b8 = (u32)__builtin_ctz(xw | 0x80000000u) & 24u;
                const u32 rk = np + __builtin_amdgcn_mbcnt_hi((u32)(m >> 32), __builtin_amdgcn_mbcnt_lo((u32)m, 0u));
                list[has ? rk : 255u] = ((i * 10u + ws * 4u + (b8 >> 3)) << 8) | ((dw >> b8) & 0xffu);
                const u32 keep = ~(0xffu << b8);
                x[0] &= ws == 0u ? keep : ~0u; x[1] &= ws == 1u ? keep : ~0u; x[2] &= ws == 2u ? keep : ~0u;
                np += __builtin_popcountll(m);
            }
        }
        asm volatile("" ::: "memory");
        raw = np > 127u || nsw + np > 128u;
        const u32 l0 = list[(lane - nsw) & 127u], l1 = list[(lane + 64u - nsw) & 127u];
        if ((u32)lane >= nsw) t0 = l0;
        if (lane + 64u >= nsw) t1 = l1;
        asm volatile("" ::: "memory");
    }
    before_stores();
    if (!raw) {
        __builtin_amdgcn_raw_buffer_store_b128((u32x4){L[0], L[1], L[2], L[3]}, dst, slot + lane * 16, 0, 17);
        __builtin_amdgcn_raw_buffer_store_b128((u32x4){L[4], L[5], L[6], L[7]}, dst, slot + 1024 + lane * 16, 0, 17);
        __builtin_amdgcn_raw_buffer_store_b96((u32x3){C[0], C[1], t0}, dst, slot + 2048 + lane * 12, 0, 17);
        if (nsw + np > 64u) __builtin_amdgcn_raw_buffer_store_b32(t1, dst, slot + 2816 + lane * 4, 0, 17);
        return win | (nsw << 15) | (np << 23);
    }
#pragma unroll
    for (int j = 0; j < 4; ++j) __builtin_amdgcn_raw_buffer_store_b128(r[j], dst, slot + j * 1024 + lane * 16, 0, 17);
    return 0x80000000u;
}

__device__ __forceinline__ void unpack_unit(u32x4 (&o)[4], u32x4 la, u32x4 lb, u32x3 ct, u32 t1, u32 meta, LAS char* st, int lane) {
    const u32 B2 = ((meta & 0xfffu) << 4) * 0x10001u, nsw = (meta >> 15) & 0xffu, np = (meta >> 23) & 0x7fu;
    const u32 L[8] = {la[0], la[1], la[2], la[3], lb[0], lb[1], lb[2], lb[3]};
    const u32 C[2] = {ct[0], ct[1]};
    auto expand = [&](u32 w, u32 i) {
        const u32 t = (w >> 6) & 0x03030303u;
        const u32 d0 = sym_to_d(w & 0x07070707u);
        const u32 d1 = sym_to_d((w >> 3) & 0x07070707u);
        const u32 d2 = sym_to_d(t | (t >> 14));
        volatile LAS unsigned short* q = (volatile LAS unsigned short*)(st + i * 10u);
        q[0] = (unsigned short)d0; q[1] = (unsigned short)(d0 >> 16); q[2] = (unsigned short)d1; q[3] = (unsigned short)(d1 >> 16);
        q[4] = (unsigned short)d2;
    };
    auto patch = [&](u32 w, u32 i) {
        *(volatile LAS unsigned char*)(st + (i - nsw < np ? (w >> 8) & 0x7ffu : 2304u)) = (unsigned char)w;
    };
    expand(ct[2], lane);
    if (nsw + np > 64u) { expand(t1, lane + 64u); patch(t1, lane + 64u); }
    patch(ct[2], lane);
    asm volatile("" ::: "memory");
    const u32 E[2] = {C[0] & (C[0] >> 1) & 0x55555555u, C[1] & (C[1] >> 1) & 0x55555555u};
    const u32 tot = __builtin_popcount(E[0]) + __builtin_popcount(E[1]);
    u32 p = wave_scan(tot) - tot + (u32)(size_t)st;
    u32 s0[8], s1[8], sel[8];
#pragma unroll
    for (int k = 0; k < 8; ++k) {
        const u32 f = (E[k >> 2] >> (2 * (k & 3))) & 0x01010101u;
        const LAS u32* q = (const LAS u32*)(size_t)(p & ~3u);
        s0[k] = q[0]; s1[k] = q[1];
        sel[k] = (f << 8) | (p & 3u);
        p += __builtin_popcount(f);
    }
    SCHED();
#pragma unroll
    for (int k = 0; k < 8; ++k) {
        const u32 c = (C[k >> 2] >> (2 * (k & 3))) & 0x03030303u;
        const u32 d = bfi(perm(0u, 0xff000000u, c), perm(s1[k], s0[k], sel[k] * 0x01010101u), c);
        const u32 x0 = pk_add(perm(d, L[k], 0x05010400u), B2), x1 = pk_add(perm(d, L[k], 0x07030602u), B2);
        o[k >> 1][(k & 1) * 2] = __builtin_amdgcn_alignbit(x0, x0, 1);
        o[k >> 1][(k & 1) * 2 + 1] = __builtin_amdgcn_alignbit(x1, x1, 1);
    }
    asm volatile("" ::: "memory");
}

struct Job { u32 hdr, src, dst, flag; bool out; };

template <int A, int B, int L, int N>
__device__ __forceinline__ bool order(u32 n, u32& s, u32& r) {
    if (n < A * L) { s = n / A; r = n % A; return true; }
    n -= A * L;
    if (n < (N - L) * (A + B)) {
        s = n / (A + B); r = n % (A + B);
        if (r < B) return false;
        s += L; r -= B; return true;
    }
    n -= (N - L) * (A + B);
    s = N - L + n / B; r = n % B;
    return false;
}
constexpr u32 U_INTRA = NSTEP * KEPT, U_CROSS_IN = NSTEP * (F + 8), U_CROSS = U_CROSS_IN + (128 - F) * 4;

__device__ __forceinline__ Job decode(u32 n, bool cross, u32 p, u32 rank, u32 hdr_par) {
    Job J;
    J.out = false;
    u32 s, r, t;
    if (cross && n >= U_CROSS_IN - MIX_STEPS * F && n < U_CROSS_IN + MIX_STEPS * MIX_OUT) {
        const u32 m = n - (U_CROSS_IN - MIX_STEPS * F), q = m / (F + MIX_OUT), e = m % (F + MIX_OUT);
        n = e < MIX_OUT ? U_CROSS_IN + q * MIX_OUT + e : U_CROSS_IN - MIX_STEPS * F + q * F + (e - MIX_OUT);
    }
    if (!cross) {
        const bool a = order<KA, KEPT - KA, LEAD_I, NSTEP>(n, s, r);
        t = (a ? 0 : KA) + r;
    } else if (n < U_CROSS_IN) {
        const bool a = order<8, F, LEAD_C, NSTEP>(n, s, r);
        t = KEPT + r;
        if (a) {
            J.hdr = hdr_par + HDR_W + ((p & 3) * 1024 + s * 8 + r) * 8;
            J.src = PK_W + ((s * 8 + r) * 4 + (rank & 3)) * 4096;
            J.dst = OFF_WRAW + ((s * 4 + (p & 3)) * 8 + r) * 4096;
            J.flag = OFF_WFLAG + (((p & 3) * 128 + s) * 8 + r) * 4;
            return J;
        }
    } else {
        n -= U_CROSS_IN;
        u32 i = n >> 2, gs;
        constexpr u32 KB = KEPT - KA, TO = KEPT / 5, TA = 3 * KA / 5, TB = 3 * KB / 5;
        static_assert(KA % 5 == 0 && KEPT % 5 == 0, "classes are whole tiles");
        if (AGK_XLAST && i >= KEPT + 3 * KA) i = i < KEPT + 3 * KA + 3 * KB ? i + 3 * F : i - 3 * KB;
        if (i < KEPT) gs = p * 32 + (i % TO) * 5 + i / TO;
        else if ((i -= KEPT) < 3 * KA) gs = ((p ^ (1 + (i % TA) % 3)) & 7u) * 32 + ((i % TA) / 3) * 5 + i / TA;
        else if ((i -= 3 * KA) < 3 * F) {
            const u32 k = i / 3;
            gs = ((rank ^ (1 + i % 3)) & 7u) * 32 + KEPT + (k < 4 ? (k & 1) * 5 + (k >> 1) : k < 4 + F - 10 ? k + 6 : ((k - (F - 10)) & 1) * 5 + ((k - (F - 10)) >> 1));
        }
        else { i -= 3 * F; gs = ((p ^ (1 + (i % TB) % 3)) & 7u) * 32 + KA + ((i % TB) / 3) * 5 + i / TB; }
        gs &= 255u;
        const u32 e = gs * 4 + (n & 3);
        J.hdr = hdr_par + HDR_O + ((p & 3) * 1024 + e) * 8;
        J.src = PK_O + ((rank & 3) * 1024 + e) * 4096;
        J.dst = e * 8 * 2048 + (p & 3) * 512;
        J.flag = 0xfffffff0u;
        J.out = true;
        return J;
    }
    s &= 127u;
    const u32 gs = p * 32 + t;
    J.hdr = hdr_par + HDR_X + (gs * 128 + s) * 8;
    J.src = PK_X + (s * 32 + t) * 4096;
    J.dst = OFF_XRAW + (s * 256 + gs) * 4096;
    J.flag = OFF_XFLAG + (gs * 128 + s) * 4;
    return J;
}

__device__ __forceinline__ void pull_link(char* ws, char* out, const char* peer_pk, u32 dir, u32 p, u32 rank, u32 nwaves,
                                          u32 hdr_par, u32 call, int lane, LAS char* st, u64* dbg) {
    const bool cross = dir >= 4;
    const u32 U = cross ? U_CROSS : U_INTRA;
    const auto ctrl = mkbuf(ws, CTRL_BYTES);
    const auto src = mkbuf(peer_pk, PK_PAR);
    u32* const tmo = (u32*)(ws + OFF_TMO);
    const u32 tick_off = lane == 0 ? (u32)OFF_TICKET + dir * 128 : 0xfffffff0u;
    const u32 tbase = (call - 1u) * (U + 2u * nwaves);
    auto set_flag = [&](u32 off) { __builtin_amdgcn_raw_buffer_store_b32(call, ctrl, lane == 0 ? off : 0xfffffff0u, 0, 17); };
    u32 t1 = uni(__builtin_amdgcn_raw_ptr_buffer_atomic_add_i32(1, ctrl, tick_off, 0, 0)) - tbase;
    u32 tk = __builtin_amdgcn_raw_ptr_buffer_atomic_add_i32(1, ctrl, tick_off, 0, 0);
    Job J1 = decode(t1, cross, p, rank, hdr_par), J0 = J1;
    u32x2 g = {0u, 0u};
    u32x4 la = {0u, 0u, 0u, 0u}, lb = la;
    u32x3 ct = {0u, 0u, 0u};
    asm volatile("" : "+v"(tk));
    u32 tx = 0, meta0 = 0, pend = 0xfffffff0u;
    bool have0 = false, dead0 = false, gone = false;
    for (;;) {
        u32x4 ca = la, cb = lb;
        u32x3 cc = ct;
        u32 cx = tx;
        asm volatile("" : "+v"(ca), "+v"(cb), "+v"(cc), "+v"(cx));
        u32 meta1 = uni(g[0]);
        const bool more = t1 < U, valid = uni(g[1]) == call;
        const u32 t2 = uni(tk) - tbase;
        bool dead1 = false;
        Job J2;
        auto issue = [&]() {
            const bool raw = (meta1 >> 31) != 0u || dead1;
            const u32 ntail = ((meta1 >> 15) & 0xffu) + ((meta1 >> 23) & 0x7fu);
            asm volatile("" ::: "memory");
            la = __builtin_amdgcn_raw_buffer_load_b128(src, lane * 16, J1.src, 17);
            lb = __builtin_amdgcn_raw_buffer_load_b128(src, lane * 16 + 1024, J1.src, 17);
            ct = __builtin_amdgcn_raw_buffer_load_b96(src, raw ? 0x7ffffff0u : lane * 12 + 2048, J1.src, 17);
            tx = __builtin_amdgcn_raw_buffer_load_b32(src, lane + 64u < ntail ? lane * 4 + 2816 : 0x7ffffff0u, J1.src, 17);
            tk = __builtin_amdgcn_raw_ptr_buffer_atomic_add_i32(1, ctrl, tick_off, 0, 0);
            J2 = decode(t2, cross, p, rank, hdr_par);
            g = __builtin_amdgcn_raw_buffer_load_b64(ctrl, 0, J2.hdr, 17);
        };
        auto finish0 = [&](bool packed) {
            const auto dst = mkbuf(J0.out ? out : ws, J0.out ? (u32)(16 * MiB) : (u32)OFF_PK);
            const u32 dv = J0.out ? (lane >> 5) * 2048 + (lane & 31) * 16 : lane * 16, dj = J0.out ? 4096u : 1024u;
            if (packed) {
                u32x4 o[4];
                unpack_unit(o, ca, cb, cc, cx, meta0, st, lane);
#pragma unroll
                for (int j = 0; j < 4; ++j) __builtin_amdgcn_raw_buffer_store_b128(o[j], dst, dv, J0.dst + j * dj, 17);
            } else {
                const u32 nm = dead0 ? 0x7fc07fc0u : 0u;
                const u32x4 lc = __builtin_amdgcn_raw_buffer_load_b128(src, lane * 16 + 2048, J0.src, 17);
                const u32x4 ld = __builtin_amdgcn_raw_buffer_load_b128(src, lane * 16 + 3072, J0.src, 17);
                __builtin_amdgcn_raw_buffer_store_b128(ca | nm, dst, dv, J0.dst, 17);
                __builtin_amdgcn_raw_buffer_store_b128(cb | nm, dst, dv, J0.dst + dj, 17);
                __builtin_amdgcn_raw_buffer_store_b128(lc | nm, dst, dv, J0.dst + 2 * dj, 17);
                __builtin_amdgcn_raw_buffer_store_b128(ld | nm, dst, dv, J0.dst + 3 * dj, 17);
            }
            asm volatile("" ::: "memory");
        };
        const bool packed0 = (meta0 >> 31) == 0u && !dead0;
        if (__builtin_expect(more && valid && have0 && packed0, 1)) {
            issue();
            WAIT_VM(6);
            set_flag(pend);
            finish0(true);
            pend = J0.flag;
        } else {
            WAIT_VM(0);
            set_flag(pend);
            pend = 0xfffffff0u;
            if (have0) { finish0(packed0); WAIT_VM(0); set_flag(J0.flag); }
            if (!more) break;
            const u64 tw = __builtin_amdgcn_s_memrealtime();
            for (u32 it = 1;; ++it) {
                asm volatile("" ::: "memory");
                g = __builtin_amdgcn_raw_buffer_load_b64(ctrl, 0, J1.hdr, 17);
                meta1 = uni(g[0]);
                if (uni(g[1]) == call) break;
                if ((gone || (it & 63u) == 0u) && give_up(tmo, call, tw)) { dead1 = gone = true; break; }
                __builtin_amdgcn_s_sleep(1);
            }
            issue();
            asm volatile("" : "+v"(la), "+v"(lb), "+v"(ct), "+v"(tx), "+v"(tk), "+v"(g) :: "memory");
        }
        J0 = J1; meta0 = meta1; dead0 = dead1; have0 = true;
        J1 = J2; t1 = t2;
    }
}

#define AGK_GLDS_BITS " sc0 sc1"
__device__ __forceinline__ void glds16(const char* sbase, u32 voff, u32 lds_addr) {
    asm volatile("s_mov_b32 m0, %2\n\ts_nop 0\n\tglobal_load_lds_dwordx4 %0, %1" AGK_GLDS_BITS
                 :: "v"(voff), "s"(sbase), "s"(lds_addr) : "memory");
}
__device__ __forceinline__ void glds16_half(const char* sbase, u32 voff, u32 lds_addr) {
    asm volatile("s_mov_b32 m0, %2\n\ts_mov_b32 exec_hi, 0\n\tglobal_load_lds_dwordx4 %0, %1" AGK_GLDS_BITS "\n\ts_mov_b32 exec_hi, -1"
                 :: "v"(voff), "s"(sbase), "s"(lds_addr) : "memory");
}

__device__ __forceinline__ u32 silu2(float y0, float y1) {
    f32x2 c = {__builtin_amdgcn_fmed3f(y0, -60.f, 60.f), __builtin_amdgcn_fmed3f(y1, -60.f, 60.f)};
    c *= -1.44269504088896f;
    f32x2 d = {__builtin_amdgcn_exp2f(c[0]), __builtin_amdgcn_exp2f(c[1])};
    d += 1.f;
    f32x2 r = {__builtin_amdgcn_rcpf(d[0]), __builtin_amdgcn_rcpf(d[1])};
    r *= (f32x2){y0, y1};
    return __builtin_bit_cast(u32, __builtin_convertvector(r, bf16x2));
}

__global__ void __launch_bounds__(576) ag_gemm_silu(const Params P) {
    __shared__ __attribute__((aligned(16))) char lds_raw[LDS_BYTES];
    LAS char* lds = (LAS char*)lds_raw;
    const int lane = threadIdx.x & 63, wid = uni(threadIdx.x >> 6);
    const int rank = P.rank, grp = rank & 4, oth = grp ^ 4;
    char* const ws = P.self;
    const u64 tile = P.tile[blockIdx.x];
    const u32 cb = (u32)tile & 0xffu, ridx = (u32)(tile >> 56);
    u32* const tmo = (u32*)(ws + OFF_TMO);
    STAMP(0);
    u32* const ctr = (u32*)(ws + OFF_CTR) + blockIdx.x * 16 + wid;
    const u32 ctr_v = __hip_atomic_load(ctr, __ATOMIC_RELAXED, __HIP_MEMORY_SCOPE_AGENT);
    u32 job = NJOBS, jsrc = 0, jstep = 0, jpiece = 0, jslot = 0, jslot_step = 0, jhdr = 0, jn = 0, jkind = 0;
    auto job_setup = [&]() {
        u32 c, r, u0 = 0;
        bool a = true;
        jn = job < HEAD_JOBS ? 1 : 4;
        if (job < HEAD_JOBS) { const u32 s = job / (KA + 32); r = job % (KA + 32); c = s >> 2; u0 = s & 3;
                               if (r >= KA) { const u32 i = (s & 3) * 8 + ((r - KA) >> 2); u0 = i & 3; r = KA + ((i >> 2) << 2) + ((r - KA) & 3); } }
        else a = order<KA + 32, 32 - KA, 4, NCHUNK>(job - HEAD_JOBS + HEAD_STEPS / 4 * (KA + 32), c, r);
        if (!a || r < KA) {
            const u32 t = a ? r : KA + r, s = c * 4 + u0;
            jsrc = (t * 32 + (lane >> 3)) * 16384 + s * 128 + (lane & 7) * 16; jstep = 128; jpiece = 131072;
            jslot = PK_X + (s * 32 + t) * 4096; jslot_step = 32 * 4096;
            jhdr = HDR_X + ((rank * 32 + t) * 128 + s) * 8;
            jkind = t < KEPT ? 4 : 5;
        } else {
            const u32 q = (r - KA) & 3, e = (c * 8 + ((r - KA) >> 2)) * 4 + u0;
            jsrc = (e * 8 + (lane >> 5)) * 2048 + q * 512 + (lane & 31) * 16; jstep = 16384; jpiece = 4096;
            jslot = PK_W + (e * 4 + q) * 4096; jslot_step = 4 * 4096;
            jhdr = HDR_W + ((rank & 3) * 1024 + e) * 8;
            jkind = q;
        }
    };
    u32x4 nx[4];
    auto job_load = [&]() {
        const auto in = mkbuf(jkind >= 4 ? P.x : P.w, 16 * MiB);
#pragma unroll
        for (int j = 0; j < 4; ++j) nx[j] = __builtin_amdgcn_raw_buffer_load_b128(in, jsrc, j * jpiece, 0);
        jsrc += jstep;
    };
    const u32 gw = wid * NCOMM + ridx;
    if (cb != 0xffu || gw >= PER * 7) {
        const u32 nr = P.n_remote;
        job = cb == 0xffu ? HEAD_JOBS + (NGEMM - nr) * 8 + gw - PER * 7 : ridx < nr ? wid * nr + ridx : HEAD_JOBS + wid * (NGEMM - nr) + (ridx - nr);
        if (cb != 0xffu && (wid == 8 || (ridx < nr && job >= HEAD_JOBS))) job = NJOBS;
        if (job < NJOBS) {
            if (job >= HEAD_JOBS) for (u32 i = 0; i < 4 + (job - HEAD_JOBS) / 80; ++i) __builtin_amdgcn_s_sleep(8);
            job_setup();
            job_load();
        }
    }
    const u32 call = uni(ctr_v) + 1u;
    if (lane == 0) __hip_atomic_store(ctr, call, __ATOMIC_RELAXED, __HIP_MEMORY_SCOPE_AGENT);
    const u32 par = call & 1u;
    const size_t hdr_par = OFF_HDR + par * HDR_PAR;

    if (cb == 0xffu && gw < PER * 7) {
        const u32 dir = gw % 7 + 1;
        const u32 p = (rank ^ dir) & 7u;
        pull_link(ws, P.out, P.ws[p] + OFF_PK + par * PK_PAR, dir, p, rank, PER, (u32)hdr_par, call, lane,
                  lds + wid * CODEC_BYTES,
                  (u64*)(ws + OFF_DBG) + (blockIdx.x * 8 + wid) * 8);
        if (dir < 4) STAMP(1); else STAMP(2);
        return;
    }

    const auto pk = mkbuf(ws + OFF_PK + par * PK_PAR, PK_PAR);
    const u32 ctab = compact_table(lane);
    LAS char* const st = lds + LDS_CODEC + wid * CODEC_BYTES;
    while (job < NJOBS) {
        const u32 win = choose_window(nx[0]);
        u32 hd = 0;
        for (u32 u = 0; u < jn; ++u) {
            const u32x4 r[4] = {nx[0], nx[1], nx[2], nx[3]};
            if (u + 1 < jn) job_load();
            const u32 meta = pack_unit(r, win, ctab, st, pk, jslot + u * jslot_step, lane,
                                       [&]() { asm volatile("" : "+v"(nx[0]), "+v"(nx[1]), "+v"(nx[2]), "+v"(nx[3])); });
            if ((u32)(lane & 3) == u) hd = meta;
        }
        char* b = nullptr;
#pragma unroll
        for (int q = 0; q < 4; ++q) {
            char* pq = P.ws[(jkind == 4 ? rank ^ (q < 3 ? q + 1 : 1) : jkind == 5 ? oth | q : oth | jkind) & 7];
            if ((lane >> 2) == q) b = pq;
        }
        const bool on = (u32)(lane & 3) < jn && (u32)(lane >> 2) < (jkind == 4 ? 3u : jkind == 5 ? 4u : 1u);
        u64* const hp = (u64*)(b + hdr_par + jhdr) + (lane & 3);
        job = cb == 0xffu ? job + NSPARE : NJOBS;
        WAIT_VM(0);
        if (on) __hip_atomic_store(hp, ((u64)call << 32) | hd, __ATOMIC_RELAXED, __HIP_MEMORY_SCOPE_SYSTEM);
        if (job < NJOBS) { job_setup(); job_load(); }
    }
    if (cb == 0xffu) { STAMP(7); return; }
    STAMP(1);

    const int wm = wid >> 2, wn = wid & 3, fr = lane & 15, fq = lane >> 4;
    const u32 ns = (u32)(tile >> 8) & 0xffu;
    auto strip_of = [&](u32 i) -> u32 { return (u32)(tile >> (16 + 8 * i)) & 0xffu; };
    if (wid == 8) {
        const u32 i = lane & 15, jj = lane >> 4;
        const u32 gs = strip_of(i < 5 ? i : 0);
        const u32 off = i < 5 ? OFF_XFLAG + (gs * 128 + jj) * 4 : OFF_WFLAG + (((cb & 3) * 128 + jj) * 8 + ((i - 5) & 7)) * 4;
        const u32 sh = i < 5 ? 2 : 5;
        const u64 skip = __builtin_amdgcn_ballot_w64(i < 5 ? (gs >> 5) == (u32)rank : (cb < 4 || i >= 13));
        const auto ctrl = mkbuf(ws, CTRL_BYTES);
        u32 ready = skip == ~0ull ? NSTEP : 0;
        for (u32 s = 0; s < NSTEP; ++s) {
            if (s >= ready) {
                const u64 t0 = __builtin_amdgcn_s_memrealtime();
                for (u32 it = 1;; ++it) {
                    asm volatile("" ::: "memory");
                    const u32 v = __builtin_amdgcn_raw_buffer_load_b32(ctrl, off + (s << sh), 0, 17);
                    const u64 m = ~(__builtin_amdgcn_ballot_w64(v == call) | skip);
                    ready = s + (m == 0ull ? 4u : (u32)__builtin_ctzll(m) >> 4);
                    if (s < ready) break;
                    if ((it & 63u) == 0u && give_up(tmo, call, t0)) { WAIT_VM(0); ready = NSTEP; break; }
                    __builtin_amdgcn_s_sleep(AGK_PSLEEP);
                }
            }
            asm volatile("" ::: "memory");
            __builtin_amdgcn_s_barrier();
            if (s == 2) __builtin_amdgcn_s_barrier();
        }
        return;
    }
    const u32 s01 = strip_of((0x31142200u >> (4 * wid)) & 7u), h01 = (0x4au >> wid) & 1u, s2 = strip_of(wid < 4 ? 3 : 4);
    const bool own01 = (s01 >> 5) == (u32)rank, own2 = (s2 >> 5) == (u32)rank;
    const char* gA01 = own01 ? P.x + (size_t)(s01 & 31) * (32 * 16384) : ws + OFF_XRAW + (size_t)s01 * 4096;
    const char* gA2 = own2 ? P.x + (size_t)(s2 & 31) * (32 * 16384) : ws + OFF_XRAW + (size_t)s2 * 4096;
    const char* gB = cb < 4 ? P.w + cb * 512 : ws + OFF_WRAW + (cb & 3) * 32768;
    const u32 dA01 = own01 ? 128u : 1048576u, dA2 = own2 ? 128u : 1048576u;
    const u32 rs01 = own01 ? 16384u : 128u, rs2 = own2 ? 16384u : 128u, rsB = cb < 4 ? 2048u : 512u;
    const u32 l3 = lane >> 3;
    const u32 vA0 = (h01 * 16 + l3) * rs01 + (((lane & 7) ^ (l3 >> 1)) * 16);
    const u32 vA1 = (h01 * 16 + 8 + l3) * rs01 + (((lane & 7) ^ (4 + (l3 >> 1))) * 16);
    const u32 vA2 = (16 + 4 * (wid & 3) + (l3 & 3)) * rs2 + (((lane & 7) ^ (((4 * wid + (l3 & 3)) >> 1) & 7)) * 16);
    u32 vB[2];
#pragma unroll
    for (int i = 0; i < 2; ++i) {
        const u32 k = 8 * wid + 2 * i + (lane >> 5), f = (k & 3) | (((k >> 3) & 1) << 2);
        vB[i] = k * rsB + (((lane & 31) ^ (f << 1)) * 16);
    }
    const u32 lds0 = (u32)(size_t)lds;
    const u32 rA0 = (wm * 80 + fr) * 128 + ((fq ^ (fr >> 1)) * 16), rA1 = rA0 ^ 64;
    u32 rB[4];
    {
        const u32 q = fr >> 2, p4 = fr & 3, f = q | ((fq & 1) << 2);
#pragma unroll
        for (int n = 0; n < 4; ++n) rB[n] = A_BYTES + (8 * fq + q) * 512 + (((wn * 4 + n) ^ f) * 32) + p4 * 8;
    }

    auto stage_piece = [&](int i, u32 slot_base) {
        if (i == 0) glds16(gA01, vA0, slot_base + wid * 2048);
        else if (i == 1) glds16(gA01, vA1, slot_base + wid * 2048 + 1024);
        else if (i == 2) glds16_half(gA2, vA2, slot_base + 16384 + wid * 512);
        else glds16(gB + (i >= 5 ? 4 * rsB : 0u), vB[(i - 3) & 1], slot_base + A_BYTES + wid * 4096 + (i - 3) * 1024);
    };
    auto advance = [&]() { gA01 += dA01; gA2 += dA2; gB += 131072; };

    f32x4 acc[5][4];
#pragma unroll
    for (int m = 0; m < 5; ++m)
#pragma unroll
        for (int n = 0; n < 4; ++n) acc[m][n] = (f32x4){0.f, 0.f, 0.f, 0.f};
    s16x8 fa[2][5];
    s16x4 fb[2][4][2];
    auto frag_read = [&](int set, int i, u32 sb, int kk) {
        if (i == 2) fa[set][0] = *(const LAS s16x8*)(lds + sb + (kk ? rA1 : rA0));
        else if (i >= 9) fa[set][i - 8] = *(const LAS s16x8*)(lds + sb + (kk ? rA1 : rA0) + (i - 8) * 2048);
        else {
            const int b = i < 2 ? i : i - 1, n = b >> 1, t = b & 1;
            fb[set][n][t] = __builtin_amdgcn_ds_read_tr16_b64_v4i16((LAS s16x4*)(lds + sb + rB[n] + t * 2048 + kk * 16384));
        }
    };
    auto mma = [&](int set, int i) {
        const int m = i >> 2, n = i & 3;
        const s16x8 b = __builtin_shufflevector(fb[set][n][0], fb[set][n][1], 0, 1, 2, 3, 4, 5, 6, 7);
        acc[m][n] = __builtin_amdgcn_mfma_f32_16x16x32_bf16(__builtin_bit_cast(bf16x8, b), __builtin_bit_cast(bf16x8, fa[set][m]), acc[m][n], 0, 0, 0);
    };

#pragma unroll
    for (int s = 0; s < 3; ++s) {
        __builtin_amdgcn_s_barrier();
        if (s == 0) STAMP(2);
#pragma unroll
        for (int i = 0; i < 7; ++i) stage_piece(i, lds0 + s * STAGE);
        advance();
    }
    WAIT_VM(14);
    __builtin_amdgcn_s_barrier();
    asm volatile("" ::: "memory");
#pragma unroll
    for (int i = 0; i < 13; ++i) frag_read(0, i, 0, 0);
    u32 cur = 0;
#define KSTEP(LOAD, VMW, NEXT) do { \
        const u32 nxt = cur == 2 * STAGE ? 0u : cur + STAGE; \
        _Pragma("unroll") for (int i = 0; i < 20; ++i) { mma(0, i); SCHED(); if (i < 13) frag_read(1, i, cur, 1); SCHED(); } \
        if (NEXT) WAIT_VM(VMW); \
        WAIT_LGKM0(); \
        if (NEXT) { __builtin_amdgcn_s_barrier(); asm volatile("" ::: "memory"); } \
        SCHED(); \
        _Pragma("unroll") for (int i = 0; i < 20; ++i) { \
            mma(1, i); SCHED(); \
            if (NEXT && i < 13) frag_read(0, i, nxt, 0); \
            if (LOAD && i >= 13) stage_piece(i - 13, lds0 + cur); \
            SCHED(); } \
        if (LOAD) advance(); \
        cur = nxt; } while (0)
    for (u32 t = 0; t < NSTEP - 3; ++t) KSTEP(true, 7, true);
    const u32 tmo_v = __hip_atomic_load(tmo, __ATOMIC_RELAXED, __HIP_MEMORY_SCOPE_AGENT);
    { const u32 t = 0; (void)t; KSTEP(false, 7, true); KSTEP(false, 0, true); KSTEP(false, 0, false); }
#undef KSTEP
    STAMP(3);

    const u32 nanm = tmo_v == call ? 0x7fc07fc0u : 0u;
    int tz = threadIdx.x;
    asm volatile("" : "+v"(tz));
    const int ln = tz & 63;
    LAS char* const img = lds + (wm * 32 + (ln & 15)) * EP_STRIDE + (wn * 64 + (ln >> 4) * 4) * 2;
    auto act = [&](int m) {
        LAS char* q = img + (m < 4 ? (m >> 1) * 64 + (m & 1) * 16 : 128 - wm * 16) * EP_STRIDE;
#pragma unroll
        for (int n = 0; n < 4; ++n) {
            const f32x4 a = acc[m][n];
            *(LAS u32x2*)(q + n * 32) = (u32x2){silu2(a[0], a[1]) | nanm, silu2(a[2], a[3]) | nanm};
        }
    };
    const auto out = mkbuf(P.out, 16 * MiB);
    const u32 q4 = cb & 3, ctab2 = compact_table(ln);
    u64* const hbase = (u64*)(P.ws[(oth | q4) & 7] + hdr_par + HDR_O) + (rank & 3) * 1024;
    u32 win = 0;
    __builtin_amdgcn_s_barrier();
    asm volatile("" ::: "memory");
    act(0);
    for (u32 ph = 0; ph < 3; ++ph) {
        if (ph == 0) act(1);
        if (ph == 1) act(3);
        WAIT_LGKM0();
        __builtin_amdgcn_s_barrier();
        asm volatile("" ::: "memory");
        const u32 u = ph * 8 + wid, e = strip_of(u >> 2) * 4 + (u & 3);
        const bool on = u < ns * 4;
        u32 meta = 0;
        if (on) {
            u32x4 r[4];
#pragma unroll
            for (int j = 0; j < 4; ++j) r[j] = *(const LAS u32x4*)(lds + (u * 8 + 2 * j + (ln >> 5)) * EP_STRIDE + (ln & 31) * 16);
            if (cb < 4) {
#pragma unroll
                for (int j = 0; j < 4; ++j)
                    __builtin_amdgcn_raw_buffer_store_b128(r[j], out, (ln >> 5) * 2048 + (ln & 31) * 16, e * 16384 + j * 4096 + cb * 512, 16);
            } else {
                if (ph == 0) win = choose_window(r[0]);
                meta = pack_unit(r, win, ctab2, st, pk, PK_O + (q4 * 1024 + e) * 4096, ln, []() {});
            }
        }
        asm volatile("" ::: "memory");
        if (ph == 0) act(2);
        if (ph == 1) act(4);
        if (on && cb >= 4) {
            WAIT_VM(0);
            if (ln == 0) __hip_atomic_store(hbase + e, ((u64)call << 32) | meta, __ATOMIC_RELAXED, __HIP_MEMORY_SCOPE_SYSTEM);
        }
    }
    STAMP(4);
}

static void build_tiles(int rank, u64* tile, int* n_early = nullptr, int* n_remote = nullptr) {
    const int grp = rank & 4, oth = grp ^ 4;
    u64 rows[8][32];
    int key[8][32], nrow[8];
    for (int cb = 0; cb < 8; ++cb) {
        int q[160], nq = 0;
        for (int t = 0; t < KEPT; ++t) q[nq++] = rank * 32 + t;
        for (int d = 1; d < 4; ++d) for (int t = 0; t < KA; ++t) q[nq++] = (rank ^ d) * 32 + t;
        for (int d = 1; d < 4; ++d) for (int t = KA; t < KEPT; ++t) q[nq++] = (rank ^ d) * 32 + t;
        for (int i = 0; i < 4; ++i) if (cb < 4 || (cb & 3) != i) for (int t = KEPT; t < KEPT + 10; ++t) q[nq++] = (oth | i) * 32 + t;
        for (int t = KEPT + 10; t < 32; ++t) for (int i = 0; i < 4; ++i) if (cb < 4 || (cb & 3) != i) q[nq++] = (oth | i) * 32 + t;
        if (cb < 4) for (int t = KEPT; t < 32; ++t) q[nq++] = rank * 32 + t;
        nrow[cb] = 0;
        for (int s0 = 0; s0 < nq; s0 += 5) {
            const int n = nq - s0 > 5 ? 5 : nq - s0;
            u64 e = (u64)cb | ((u64)n << 8);
            for (int s = 0; s < 5; ++s) e |= (u64)q[s0 + (s < n ? s : 0)] << (16 + 8 * s);
            const int g = q[s0], r = nrow[cb];
            key[cb][r] = r < 4 * KEPT / 5 ? 2 + r : (g >> 5) == rank ? r & 1 : (g & 31) < KEPT + 10 ? 40 + ((g >> 5) & 3) * 2 + ((g & 31) - KEPT) / 5 : 60 + r;
            rows[cb][nrow[cb]++] = e;
        }
    }
    u64 list[8][32];
    for (int half = 0; half < 2; ++half) {
        int n = 0;
        for (int k = 0; k < 100; ++k)
            for (int c = 0; c < 4; ++c) {
                const int cb = (c >> 1) * 4 + half * 2 + (c & 1);
                for (int r = 0; r < nrow[cb]; ++r)
                    if (key[cb][r] == k) { list[half * 4 + n / (NGEMM / 8)][n % (NGEMM / 8)] = rows[cb][r]; ++n; }
            }
    }
    int ng = 0, nc = 0;
    for (int b = 0; b < NWG; ++b) {
        const int xcd = b & 7, i = b >> 3;
        tile[b] = i < NCOMM / 8 ? 0xffull | ((u64)nc++ << 56) : list[xcd][i - NCOMM / 8];
    }
    for (int cls = 0; cls < 3; ++cls) {
        for (int b = 0; b < NWG; ++b) {
            const int cb = (int)(tile[b] & 0xff), gs = (int)((tile[b] >> 16) & 0xff);
            if (cb == 0xff) continue;
            const int c = cb < 4 ? 2 : ((gs >> 5) == rank || (gs & 31) < KA) ? 0 : 1;
            if (c == cls) tile[b] |= (u64)ng++ << 56;
        }
        if (cls == 0 && n_early) *n_early = ng;
        if (cls == 1 && n_remote) *n_remote = ng;
    }
}
}

extern "C" void kernel_launch(int rank, int nranks, ncclComm_t comm,
                              hipStream_t stream,
                              void* const* d_in, const int* in_sizes, int n_in,
                              void* d_out, int out_size,
                              const int* in_dtypes, int out_dtype,
                              void* const* d_out_all,
                              void* const* d_ws_all, size_t ws_bytes) {
    using namespace agk;
    if (nranks != NRANK || (unsigned)rank >= (unsigned)NRANK || ws_bytes < WS_NEED || n_in < 2 ||
        in_sizes[0] != 1024 * KDIM || in_sizes[1] != KDIM * 1024 || out_size != 8192 * 1024) {
        fprintf(stderr, "kernel_launch: unexpected problem shape\n");
        return;
    }
    Params P;
    for (int p = 0; p < NRANK; ++p) P.ws[p] = (char*)d_ws_all[p];
    P.x = (const char*)d_in[0];
    P.w = (const char*)d_in[1];
    P.out = (char*)d_out;
    P.rank = rank;
    P.self = P.ws[rank];
    P.pad = 0;
    build_tiles(rank, P.tile, &P.n_early, &P.n_remote);
    if ((u32)P.n_remote * 8 < HEAD_JOBS) return;

    hipLaunchKernelGGL(ag_gemm_silu, dim3(NWG), dim3(576), 0, stream, P);
    HIP_CHECK(hipGetLastError());
}
```

```cpp
#include <hip/hip_runtime.h>
#include <rccl/rccl.h>
#include <cstdio>

#define HIP_CHECK(expr) do { hipError_t _e = (expr); if (_e != hipSuccess) { \
    fprintf(stderr, "HIP error %s at %s:%d\n", hipGetErrorString(_e), __FILE__, __LINE__); \
    return; } } while (0)

namespace agk {
#define LAS __attribute__((address_space(3)))
typedef unsigned u32;
typedef unsigned long long u64;
typedef unsigned u32x4 __attribute__((ext_vector_type(4)));
typedef unsigned u32x2 __attribute__((ext_vector_type(2)));
typedef short s16x4 __attribute__((ext_vector_type(4)));
typedef short s16x8 __attribute__((ext_vector_type(8)));
typedef __bf16 bf16x8 __attribute__((ext_vector_type(8)));
typedef __bf16 bf16x2 __attribute__((ext_vector_type(2)));
typedef float f32x4 __attribute__((ext_vector_type(4)));
typedef float f32x2 __attribute__((ext_vector_type(2)));

constexpr int NRANK = 8, KDIM = 8192;
#define AGK_F 12
constexpr int F = AGK_F, KEPT = 32 - F;
constexpr int NWG = 256;
#define AGK_PER 28
#define AGK_MS 0
#define AGK_MO 10
#define AGK_HS 8
#define AGK_PSLEEP 2
#define AGK_TAU 96
#define AGK_T0 1800
#define AGK_PURE 1
#define AGK_PRIO 3
#define AGK_SLOPE 80
#define AGK_DELAY0 4
#define AGK_SPSLEEP 0
#define AGK_KM 10
#define AGK_LM 14
#define AGK_SMALL 0
#define AGK_LR 2
#define AGK_KA 5
#define AGK_LI 24
#define AGK_LC 21
constexpr int PER = AGK_PER;
constexpr int KA = AGK_KA, KM = AGK_KM, KB = KEPT - KA - KM, LEAD_M = AGK_LM;
constexpr int LEAD_I = AGK_LI, LEAD_C = AGK_LC;
constexpr int MIX_STEPS = AGK_MS, MIX_OUT = AGK_MO;
constexpr int NCHUNK = 32;
constexpr int NSTEP = KDIM / 64;
constexpr u32 HEAD_STEPS = AGK_HS, HEAD_JOBS = HEAD_STEPS * 64, NJOBS = HEAD_JOBS + 2048 - HEAD_STEPS * 16;
constexpr u64 DEADLINE = 50000000ull;

constexpr size_t MiB = 1ull << 20;
constexpr size_t OFF_CTR = 0;
constexpr size_t OFF_TMO = 16384;
constexpr size_t OFF_TICKET = 32 * 1024;
constexpr size_t OFF_OSEQ = 40 * 1024;
constexpr size_t OFF_XFLAG = 64 * 1024;
constexpr size_t OFF_WFLAG = 256 * 1024;
constexpr size_t OFF_DBG = 512 * 1024;
constexpr size_t OFF_HDR = 1 * MiB;
constexpr size_t HDR_PAR = MiB / 2;
constexpr size_t HDR_X = 0;
constexpr size_t HDR_W = 256 * 1024;
constexpr size_t HDR_O = 320 * 1024;
constexpr size_t CTRL_BYTES = 4 * MiB;
constexpr size_t OFF_XRAW = CTRL_BYTES;
constexpr size_t OFF_WRAW = OFF_XRAW + 128 * MiB;
constexpr size_t OFF_PK = OFF_WRAW + 16 * MiB;
constexpr size_t PK_PAR = 48 * MiB;
constexpr u32 PK_X = 0;
constexpr u32 PK_W = 16 * MiB;
constexpr u32 PK_O = 32 * MiB;
constexpr size_t WS_NEED = OFF_PK + 2 * PK_PAR;

constexpr int A_BYTES = 160 * 128, STAGE = A_BYTES + 64 * 512, LDS_BYTES = 3 * STAGE;
constexpr int EP_STRIDE = 528;
constexpr int LDS_CODEC = 96 * 1024, CODEC_BYTES = 3072;

template <u32 V> struct Const { static constexpr u32 value = V; };

struct Params {
    char* ws[NRANK];
    const char* x;
    const char* w;
    char* out;
    char* self;
    int rank, n_gemm, n_remote, pad;
    u64 tile[NWG];
};

#define STAMP(slot) do {} while (0)

#define WAIT_VM(n) asm volatile("s_waitcnt vmcnt(" #n ")" ::: "memory")
#define WAIT_LGKM0() do { asm volatile("" ::: "memory"); __builtin_amdgcn_s_waitcnt(0xC07F); asm volatile("" ::: "memory"); } while (0)
#define SCHED() __builtin_amdgcn_sched_barrier(0)

__device__ __forceinline__ __amdgpu_buffer_rsrc_t mkbuf(const void* p, u32 bytes) {
    return __builtin_amdgcn_make_buffer_rsrc((void*)p, 0, bytes, 0x00020000);
}
__device__ __forceinline__ u32 uni(u32 v) { return __builtin_amdgcn_readfirstlane(v); }
__device__ __forceinline__ u32 bfi(u32 m, u32 a, u32 b) { return (a & m) | (b & ~m); }

__device__ __forceinline__ bool give_up(u32* tmo, u32 call, u64 t0) {
    if (uni(__hip_atomic_load(tmo, __ATOMIC_RELAXED, __HIP_MEMORY_SCOPE_AGENT)) == call ||
        __builtin_amdgcn_s_memrealtime() - t0 > DEADLINE) {
        __hip_atomic_store(tmo, call, __ATOMIC_RELAXED, __HIP_MEMORY_SCOPE_AGENT);
        return true;
    }
    return false;
}

typedef unsigned short u16x2 __attribute__((ext_vector_type(2)));
typedef unsigned u32x3 __attribute__((ext_vector_type(3)));
__device__ __forceinline__ u32 pk_add(u32 a, u32 b) { return __builtin_bit_cast(u32, __builtin_bit_cast(u16x2, a) + __builtin_bit_cast(u16x2, b)); }
__device__ __forceinline__ u32 pk_sub(u32 a, u32 b) { return __builtin_bit_cast(u32, __builtin_bit_cast(u16x2, a) - __builtin_bit_cast(u16x2, b)); }
__device__ __forceinline__ u32 pk_min(u32 a, u32 b) { return __builtin_bit_cast(u32, __builtin_elementwise_min(__builtin_bit_cast(u16x2, a), __builtin_bit_cast(u16x2, b))); }
__device__ __forceinline__ u32 perm(u32 hi, u32 lo, u32 sel) { return __builtin_amdgcn_perm(hi, lo, sel); }
__device__ __forceinline__ u32 wave_scan(u32 v) {
    v += (u32)__builtin_amdgcn_update_dpp(0, (int)v, 0x111, 0xf, 0xf, true);
    v += (u32)__builtin_amdgcn_update_dpp(0, (int)v, 0x112, 0xf, 0xf, true);
    v += (u32)__builtin_amdgcn_update_dpp(0, (int)v, 0x114, 0xf, 0xf, true);
    v += (u32)__builtin_amdgcn_update_dpp(0, (int)v, 0x118, 0xf, 0xf, true);
    v += (u32)__builtin_amdgcn_update_dpp(0, (int)v, 0x142, 0xa, 0xf, false);
    v += (u32)__builtin_amdgcn_update_dpp(0, (int)v, 0x143, 0xc, 0xf, false);
    return v;
}
__device__ __forceinline__ u32 sym_to_d(u32 s) { return perm(0xfffefdfcu, 0xfbfa0403u, s); }
__device__ __forceinline__ u32 compact_table(int lane) {
    u32 t = 0x0c0c0c0cu;
#pragma unroll
    for (int b = 3; b >= 0; --b) if ((lane >> b) & 1) t = (t << 8) | (u32)b;
    return t;
}
__device__ __forceinline__ u32 choose_window(u32x4 v) {
    u32 sum = 0;
#pragma unroll
    for (int i = 0; i < 4; ++i) sum = __builtin_amdgcn_udot2(__builtin_bit_cast(u16x2, v[i] & 0x7fff7fffu), (u16x2){1, 1}, sum, false);
    const u32 mean = (u32)__builtin_amdgcn_readlane(wave_scan(sum), 63) >> 9;
    return mean > 133u ? (mean - 133u) >> 3 : 0u;
}

template <class Pre>
__device__ __forceinline__ u32 pack_unit(const u32x4 (&r)[4], u32 win, u32 ctab, LAS char* st,
                                         __amdgpu_buffer_rsrc_t dst, u32 slot, int lane, Pre before_stores) {
    const u32 B2 = ((win & 0xfffu) << 4) * 0x10001u;
    u32 L[8], D[8], C[2] = {0u, 0u};
#pragma unroll
    for (int k = 0; k < 8; ++k) {
        const u32 d0 = r[k >> 1][(k & 1) * 2], d1 = r[k >> 1][(k & 1) * 2 + 1];
        const u32 x0 = pk_sub(__builtin_amdgcn_alignbit(d0, d0, 31), B2), x1 = pk_sub(__builtin_amdgcn_alignbit(d1, d1, 31), B2);
        L[k] = perm(x1, x0, 0x06040200u);
        D[k] = perm(x1, x0, 0x07050301u);
        C[k >> 2] |= perm(pk_min(x1, 0x03ff03ffu), pk_min(x0, 0x03ff03ffu), 0x07050301u) << (2 * (k & 3));
    }
    const u32 E[2] = {C[0] & (C[0] >> 1) & 0x55555555u, C[1] & (C[1] >> 1) & 0x55555555u};
    *(LAS u32x4*)(st + lane * 16) = (u32x4){0u, 0u, 0u, 0u};
    *(LAS u32x2*)(st + 1024 + lane * 8) = (u32x2){0u, 0u};
    u32 f[8], sel[8];
#pragma unroll
    for (int k = 0; k < 8; ++k) {
        f[k] = (E[k >> 2] >> (2 * (k & 3))) & 0x01010101u;
        sel[k] = (u32)__builtin_amdgcn_ds_bpermute((int)__builtin_amdgcn_udot4(f[k], 0x20100804u, 0u, false), (int)ctab);
    }
    SCHED();
    const u32 tot = __builtin_popcount(E[0]) + __builtin_popcount(E[1]);
    const u32 incl = wave_scan(tot), n = __builtin_amdgcn_readlane(incl, 63);
#pragma unroll
    for (int k = 0; k < 8; ++k) { D[k] = perm(0u, D[k], sel[k]); asm volatile("" : "+v"(D[k])); }
    const u32 nsw = (n + 9u) / 10u;
    u32 np = 0, t0 = 0, t1 = 0;
    bool raw = nsw > 128u;
    if (!raw) {
        asm volatile("" ::: "memory");
        u32 p = incl - tot + (u32)(size_t)st;
#pragma unroll
        for (int k = 0; k < 8; ++k) {
            const u64 v = (u64)D[k] << ((p & 3u) * 8u);
            LAS u32* q = (LAS u32*)(size_t)(p & ~3u);
            __hip_atomic_fetch_or(q, (u32)v, __ATOMIC_RELAXED, __HIP_MEMORY_SCOPE_WAVEFRONT);
            __hip_atomic_fetch_or(q + 1, (u32)(v >> 32), __ATOMIC_RELAXED, __HIP_MEMORY_SCOPE_WAVEFRONT);
            p += __builtin_popcount(f[k]);
        }
        asm volatile("" ::: "memory");
        *(LAS unsigned char*)(st + n + (lane & 15)) = 0xff;
        asm volatile("" ::: "memory");
        LAS u32* const list = (LAS u32*)(st + 2048);
        for (u32 i0 = 0; i0 < nsw; i0 += 64) {
            const u32 i = i0 + lane;
            const LAS u32* q = (const LAS u32*)(st + ((i * 10u) & ~3u));
            const u32 a0 = q[0], a1 = q[1], a2 = q[2], sh = (lane & 1) * 2;
            const u32 d[3] = {__builtin_amdgcn_alignbyte(a1, a0, sh), __builtin_amdgcn_alignbyte(a2, a1, sh), __builtin_amdgcn_alignbyte(0u, a2, sh)};
            u32 sy[3], x[3];
#pragma unroll
            for (int k = 0; k < 3; ++k) {
                const u32 pos = (~d[k] >> 7) & 0x01010101u;
                sy[k] = ((d[k] & 0x07070707u) + pos * 5u) & 0x07070707u;
                x[k] = sym_to_d(sy[k]) ^ d[k];
            }
            x[2] &= 0xffffu;
            if (i >= nsw) x[0] = x[1] = x[2] = 0u;
            const u32 w = sy[0] | (sy[1] << 3) | (((sy[2] & 0x0303u) | ((sy[2] & 0x0404u) << 14)) << 6);
            if (i0 == 0) t0 = w; else t1 = w;
            for (;;) {
                const bool has = (x[0] | x[1] | x[2]) != 0u;
                const u64 m = __builtin_amdgcn_ballot_w64(has);
                if (m == 0ull || np > 127u) break;
                const u32 ws = x[0] ? 0u : x[1] ? 1u : 2u;
                const u32 xw = x[0] ? x[0] : x[1] ? x[1] : x[2], dw = x[0] ? d[0] : x[1] ? d[1] : d[2];
                const u32 b8 = (u32)__builtin_ctz(xw | 0x80000000u) & 24u;
                const u32 rk = np + __builtin_amdgcn_mbcnt_hi((u32)(m >> 32), __builtin_amdgcn_mbcnt_lo((u32)m, 0u));
                list[has ? rk : 255u] = ((i * 10u + ws * 4u + (b8 >> 3)) << 8) | ((dw >> b8) & 0xffu);
                const u32 keep = ~(0xffu << b8);
                x[0] &= ws == 0u ? keep : ~0u; x[1] &= ws == 1u ? keep : ~0u; x[2] &= ws == 2u ? keep : ~0u;
                np += __builtin_popcountll(m);
            }
        }
        asm volatile("" ::: "memory");
        raw = np > 127u || nsw + np > 128u;
        const u32 l0 = list[(lane - nsw) & 127u], l1 = list[(lane + 64u - nsw) & 127u];
        if ((u32)lane >= nsw) t0 = l0;
        if (lane + 64u >= nsw) t1 = l1;
        asm volatile("" ::: "memory");
    }
    before_stores();
    if (!raw) {
        __builtin_amdgcn_raw_buffer_store_b128((u32x4){L[0], L[1], L[2], L[3]}, dst, slot + lane * 16, 0, 17);
        __builtin_amdgcn_raw_buffer_store_b128((u32x4){L[4], L[5], L[6], L[7]}, dst, slot + 1024 + lane * 16, 0, 17);
        __builtin_amdgcn_raw_buffer_store_b96((u32x3){C[0], C[1], t0}, dst, slot + 2048 + lane * 12, 0, 17);
        if (nsw + np > 64u) __builtin_amdgcn_raw_buffer_store_b32(t1, dst, slot + 2816 + lane * 4, 0, 17);
        return win | (nsw << 15) | (np << 23);
    }
#pragma unroll
    for (int j = 0; j < 4; ++j) __builtin_amdgcn_raw_buffer_store_b128(r[j], dst, slot + j * 1024 + lane * 16, 0, 17);
    return 0x80000000u;
}

__device__ __forceinline__ void unpack_unit(u32x4 (&o)[4], u32x4 la, u32x4 lb, u32x3 ct, u32 t1, u32 meta, LAS char* st, int lane) {
    const u32 B2 = ((meta & 0xfffu) << 4) * 0x10001u, nsw = (meta >> 15) & 0xffu, np = (meta >> 23) & 0x7fu;
    const u32 L[8] = {la[0], la[1], la[2], la[3], lb[0], lb[1], lb[2], lb[3]};
    const u32 C[2] = {ct[0], ct[1]};
    auto expand = [&](u32 w, u32 i) {
        const u32 t = (w >> 6) & 0x03030303u;
        const u32 d0 = sym_to_d(w & 0x07070707u);
        const u32 d1 = sym_to_d((w >> 3) & 0x07070707u);
        const u32 d2 = sym_to_d(t | (t >> 14));
        volatile LAS unsigned short* q = (volatile LAS unsigned short*)(st + i * 10u);
        q[0] = (unsigned short)d0; q[1] = (unsigned short)(d0 >> 16); q[2] = (unsigned short)d1; q[3] = (unsigned short)(d1 >> 16);
        q[4] = (unsigned short)d2;
    };
    auto patch = [&](u32 w, u32 i) {
        *(volatile LAS unsigned char*)(st + (i - nsw < np ? (w >> 8) & 0x7ffu : 2304u)) = (unsigned char)w;
    };
    expand(ct[2], lane);
    if (nsw + np > 64u) { expand(t1, lane + 64u); patch(t1, lane + 64u); }
    patch(ct[2], lane);
    asm volatile("" ::: "memory");
    const u32 E[2] = {C[0] & (C[0] >> 1) & 0x55555555u, C[1] & (C[1] >> 1) & 0x55555555u};
    const u32 tot = __builtin_popcount(E[0]) + __builtin_popcount(E[1]);
    u32 p = wave_scan(tot) - tot + (u32)(size_t)st;
    u32 s0[8], s1[8], sel[8];
#pragma unroll
    for (int k = 0; k < 8; ++k) {
        const u32 f = (E[k >> 2] >> (2 * (k & 3))) & 0x01010101u;
        const LAS u32* q = (const LAS u32*)(size_t)(p & ~3u);
        s0[k] = q[0]; s1[k] = q[1];
        sel[k] = (f << 8) | (p & 3u);
        p += __builtin_popcount(f);
    }
    SCHED();
#pragma unroll
    for (int k = 0; k < 8; ++k) {
        const u32 c = (C[k >> 2] >> (2 * (k & 3))) & 0x03030303u;
        const u32 d = bfi(perm(0u, 0xff000000u, c), perm(s1[k], s0[k], sel[k] * 0x01010101u), c);
        const u32 x0 = pk_add(perm(d, L[k], 0x05010400u), B2), x1 = pk_add(perm(d, L[k], 0x07030602u), B2);
        o[k >> 1][(k & 1) * 2] = __builtin_amdgcn_alignbit(x0, x0, 1);
        o[k >> 1][(k & 1) * 2 + 1] = __builtin_amdgcn_alignbit(x1, x1, 1);
    }
    asm volatile("" ::: "memory");
}

struct Job { u32 hdr, src, dst, flag; bool out; };

template <int A, int B, int L, int N>
__device__ __forceinline__ bool order(u32 n, u32& s, u32& r) {
    if (n < A * L) { s = n / A; r = n % A; return true; }
    n -= A * L;
    if (n < (N - L) * (A + B)) {
        s = n / (A + B); r = n % (A + B);
        if (r < B) return false;
        s += L; r -= B; return true;
    }
    n -= (N - L) * (A + B);
    s = N - L + n / B; r = n % B;
    return false;
}
constexpr u32 U_OUT = (128 - F) * 4, TAG_MASK = 0x3fffffu;
constexpr u32 U_INTRA = NSTEP * KEPT, U_CROSS_IN = NSTEP * (F + 8), U_CROSS = U_CROSS_IN + U_OUT;

__device__ __forceinline__ Job decode(u32 n, bool cross, u32 p, u32 rank, u32 hdr_par) {
    Job J;
    J.out = false;
    u32 s, r, t;
    if (cross && n >= U_CROSS_IN - MIX_STEPS * F && n < U_CROSS_IN + MIX_STEPS * MIX_OUT) {
        const u32 m = n - (U_CROSS_IN - MIX_STEPS * F), q = m / (F + MIX_OUT), e = m % (F + MIX_OUT);
        n = e < MIX_OUT ? U_CROSS_IN + q * MIX_OUT + e : U_CROSS_IN - MIX_STEPS * F + q * F + (e - MIX_OUT);
    }
    if (!cross) {
        constexpr u32 D1 = LEAD_I - LEAD_M, D2 = LEAD_I, H = HEAD_STEPS;
        constexpr u32 NH = H * KEPT, N0 = NH + D1 * KA, N1 = N0 + (D2 - D1) * (KA + KM), N2 = N1 + (NSTEP - H - D2) * KEPT, N3 = N2 + D1 * (KM + KB);
        if (n < NH) { s = n / KEPT; t = n % KEPT; }
        else {
            if (n < N0) { s = H + (n - NH) / KA; t = (n - NH) % KA; }
            else if (n < N1) { s = H + D1 + (n - N0) / (KA + KM); t = (n - N0) % (KA + KM); }
            else if (n < N2) { s = H + D2 + (n - N1) / KEPT; t = KEPT - 1 - (n - N1) % KEPT; }
            else if (n < N3) { s = NSTEP + (n - N2) / (KM + KB); t = KEPT - 1 - (n - N2) % (KM + KB); }
            else { s = NSTEP + D1 + (n - N3) / KB; t = KEPT - 1 - (n - N3) % KB; }
            s -= t < KA ? 0 : t < KA + KM ? D1 : D2;
        }
    } else if (n < U_CROSS_IN) {
        bool a;
        if (n < HEAD_STEPS * (8 + F)) a = order<8, F, 0, HEAD_STEPS>(n, s, r);
        else { a = order<8, F, LEAD_C, NSTEP - HEAD_STEPS>(n - HEAD_STEPS * (8 + F), s, r); s += HEAD_STEPS; }
        t = KEPT + r;
        if (a) {
            J.hdr = hdr_par + HDR_W + ((p & 3) * 1024 + s * 8 + r) * 8;
            J.src = PK_W + ((s * 8 + r) * 4 + (rank & 3)) * 4096;
            J.dst = OFF_WRAW + ((s * 4 + (p & 3)) * 8 + r) * 4096;
            J.flag = OFF_WFLAG + (((p & 3) * 128 + s) * 8 + r) * 4;
            return J;
        }
    } else {
        J.hdr = hdr_par + HDR_O + ((p & 3) * 1024 + ((n - U_CROSS_IN) & 1023u)) * 8;
        J.src = PK_O + (rank & 3) * 1024 * 4096;
        J.dst = (p & 3) * 512;
        J.flag = 0xfffffff0u;
        J.out = true;
        return J;
    }
    s &= 127u;
    const u32 gs = p * 32 + t;
    J.hdr = hdr_par + HDR_X + (gs * 128 + s) * 8;
    J.src = PK_X + (s * 32 + t) * 4096;
    J.dst = OFF_XRAW + (s * 256 + gs) * 4096;
    J.flag = OFF_XFLAG + (gs * 128 + s) * 4;
    return J;
}

__device__ __forceinline__ void pull_link(char* ws, char* out, const char* peer_pk, u32 dir, u32 p, u32 rank, u32 first,
                                          u32 hdr_par, u32 call, int lane, LAS char* st, u64* dbg) {
    const bool cross = dir >= 4;
    const u32 U = cross ? U_CROSS : U_INTRA;
    const auto ctrl = mkbuf(ws, CTRL_BYTES);
    const auto src = mkbuf(peer_pk, PK_PAR);
    u32* const tmo = (u32*)(ws + OFF_TMO);
    const u32 tick_off = lane == 0 ? (u32)OFF_TICKET + dir * 128 : 0xfffffff0u;
    const u32 tbase = (call - 1u) * U - 2u * PER;
    auto set_flag = [&](u32 off) { __builtin_amdgcn_raw_buffer_store_b32(call, ctrl, lane == 0 ? off : 0xfffffff0u, 0, 17); };
    __builtin_amdgcn_s_waitcnt(0x0F70);
    u32 t1 = uni(first), tk = first + PER + tbase;
    Job J1 = decode(t1, cross, p, rank, hdr_par), J0 = J1;
    u32x2 g = {0u, 0u};
    u32x4 la = {0u, 0u, 0u, 0u}, lb = la;
    u32x3 ct = {0u, 0u, 0u};
    u32 tx = 0, meta0 = 0, pend = 0xfffffff0u;
    bool have0 = false, dead0 = false, gone = false;
    for (;;) {
        u32x4 ca = la, cb = lb;
        u32x3 cc = ct;
        u32 cx = tx;
        asm volatile("" : "+v"(ca), "+v"(cb), "+v"(cc), "+v"(cx));
        u32 meta1 = uni(g[0]), tag1 = uni(g[1]);
        const bool more = t1 < U, valid = ((tag1 ^ call) & TAG_MASK) == 0u;
        const u32 t2 = uni(tk) - tbase;
        bool dead1 = false;
        Job J2;
        auto issue = [&]() {
            const bool raw = (meta1 >> 31) != 0u || dead1;
            const u32 ntail = ((meta1 >> 15) & 0xffu) + ((meta1 >> 23) & 0x7fu);
            if (J1.out) { J1.src += (tag1 >> 22) * 4096; J1.dst += (tag1 >> 22) * 16384; }
            asm volatile("" ::: "memory");
            la = __builtin_amdgcn_raw_buffer_load_b128(src, lane * 16, J1.src, 17);
            lb = __builtin_amdgcn_raw_buffer_load_b128(src, lane * 16 + 1024, J1.src, 17);
            ct = __builtin_amdgcn_raw_buffer_load_b96(src, raw ? 0x7ffffff0u : lane * 12 + 2048, J1.src, 17);
            tx = __builtin_amdgcn_raw_buffer_load_b32(src, lane + 64u < ntail ? lane * 4 + 2816 : 0x7ffffff0u, J1.src, 17);
            tk = __builtin_amdgcn_raw_ptr_buffer_atomic_add_i32(1, ctrl, tick_off, 0, 0);
            J2 = decode(t2, cross, p, rank, hdr_par);
            g = __builtin_amdgcn_raw_buffer_load_b64(ctrl, 0, J2.hdr, 17);
        };
        auto finish0 = [&](bool packed, bool take_g) {
            const auto dst = mkbuf(J0.out ? out : ws, J0.out ? (u32)(16 * MiB) : (u32)OFF_PK);
            const u32 dv = J0.dst + (J0.out ? (lane >> 5) * 2048 + (lane & 31) * 16 : lane * 16), dj = J0.out ? 4096u : 1024u;
            if (packed) {
                u32x4 o[4];
                unpack_unit(o, ca, cb, cc, cx, meta0, st, lane);
                if (take_g) asm volatile("" : "+v"(g));
#pragma unroll
                for (int j = 0; j < 4; ++j) __builtin_amdgcn_raw_buffer_store_b128(o[j], dst, dv + j * dj, 0, 17);
            } else {
                const u32 nm = dead0 ? 0x7fc07fc0u : 0u;
                asm volatile("" : "+v"(g));
                const u32x4 lc = __builtin_amdgcn_raw_buffer_load_b128(src, lane * 16 + 2048, J0.src, 17);
                const u32x4 ld = __builtin_amdgcn_raw_buffer_load_b128(src, lane * 16 + 3072, J0.src, 17);
                __builtin_amdgcn_raw_buffer_store_b128(ca | nm, dst, dv, 0, 17);
                __builtin_amdgcn_raw_buffer_store_b128(cb | nm, dst, dv + dj, 0, 17);
                __builtin_amdgcn_raw_buffer_store_b128(lc | nm, dst, dv + 2 * dj, 0, 17);
                __builtin_amdgcn_raw_buffer_store_b128(ld | nm, dst, dv + 3 * dj, 0, 17);
            }
            asm volatile("" ::: "memory");
        };
        const bool packed0 = (meta0 >> 31) == 0u && !dead0;
        if (__builtin_expect(more && valid && have0 && packed0, 1)) {
            issue();
            WAIT_VM(6);
            set_flag(pend);
            finish0(true, false);
            pend = J0.flag;
        } else {
            g = __builtin_amdgcn_raw_buffer_load_b64(ctrl, 0, J1.hdr, 17);
            WAIT_VM(1);
            set_flag(pend);
            pend = 0xfffffff0u;
            if (have0) { finish0(packed0, true); pend = J0.flag; } else asm volatile("" : "+v"(g));
            meta1 = uni(g[0]); tag1 = uni(g[1]);
            const u64 tw = __builtin_amdgcn_s_memrealtime();
            if (!more || ((tag1 ^ call) & TAG_MASK) != 0u) {
                WAIT_VM(0);
                set_flag(pend);
                pend = 0xfffffff0u;
                if (!more) break;
                for (u32 it = 1;; ++it) {
                    asm volatile("" ::: "memory");
                    g = __builtin_amdgcn_raw_buffer_load_b64(ctrl, 0, J1.hdr, 17);
                    meta1 = uni(g[0]); tag1 = uni(g[1]);
                    if (((tag1 ^ call) & TAG_MASK) == 0u) break;
                    if ((gone || (it & 63u) == 0u) && give_up(tmo, call, tw)) { dead1 = gone = true; tag1 = rank << 29; break; }
                    __builtin_amdgcn_s_sleep(1);
                }
            }
            issue();
            WAIT_VM(6);
            set_flag(pend);
            pend = 0xfffffff0u;
            asm volatile("" : "+v"(la), "+v"(lb), "+v"(ct), "+v"(tx), "+v"(tk), "+v"(g) :: "memory");
        }
        J0 = J1; meta0 = meta1; dead0 = dead1; have0 = true;
        J1 = J2; t1 = t2;
    }
}

#define AGK_GLDS_BITS " sc0 sc1"
__device__ __forceinline__ void glds16(const char* sbase, u32 voff, u32 lds_addr) {
    asm volatile("s_mov_b32 m0, %2\n\ts_nop 0\n\tglobal_load_lds_dwordx4 %0, %1" AGK_GLDS_BITS
                 :: "v"(voff), "s"(sbase), "s"(lds_addr) : "memory");
}
__device__ __forceinline__ void glds16_half(const char* sbase, u32 voff, u32 lds_addr) {
    asm volatile("s_mov_b32 m0, %2\n\ts_mov_b32 exec_hi, 0\n\tglobal_load_lds_dwordx4 %0, %1" AGK_GLDS_BITS "\n\ts_mov_b32 exec_hi, -1"
                 :: "v"(voff), "s"(sbase), "s"(lds_addr) : "memory");
}

__device__ __forceinline__ u32 silu2(float y0, float y1) {
    f32x2 c = {__builtin_amdgcn_fmed3f(y0, -60.f, 60.f), __builtin_amdgcn_fmed3f(y1, -60.f, 60.f)};
    c *= -1.44269504088896f;
    f32x2 d = {__builtin_amdgcn_exp2f(c[0]), __builtin_amdgcn_exp2f(c[1])};
    d += 1.f;
    f32x2 r = {__builtin_amdgcn_rcpf(d[0]), __builtin_amdgcn_rcpf(d[1])};
    r *= (f32x2){y0, y1};
    return __builtin_bit_cast(u32, __builtin_convertvector(r, bf16x2));
}

__global__ void __launch_bounds__(576) ag_gemm_silu(const Params P) {
    __shared__ __attribute__((aligned(16))) char lds_raw[LDS_BYTES];
    LAS char* lds = (LAS char*)lds_raw;
    const int lane = threadIdx.x & 63, wid = uni(threadIdx.x >> 6);
    const int rank = P.rank, grp = rank & 4, oth = grp ^ 4;
    char* const ws = P.self;
    const u64 tile = P.tile[blockIdx.x];
    const u32 cb = (u32)tile & 0xffu, ridx = (u32)(tile >> 56);
    u32* const tmo = (u32*)(ws + OFF_TMO);
    u32* const ctr = (u32*)(ws + OFF_CTR) + blockIdx.x * 16 + wid;
    const u32 ctr_v = __hip_atomic_load(ctr, __ATOMIC_RELAXED, __HIP_MEMORY_SCOPE_AGENT);
    u32 job = NJOBS, jsrc = 0, jstep = 0, jpiece = 0, jslot = 0, jslot_step = 0, jhdr = 0, jn = 0, jkind = 0;
    auto job_setup = [&]() {
        constexpr u32 KE = KA + KM;
        u32 c, r, u0 = 0;
        jn = job < HEAD_JOBS ? 1 : 4;
        if (job < HEAD_JOBS) {
            const u32 s = job >> 6;
            r = job & 63; c = s >> 2; u0 = s & 3;
            if (r >= 32) { const u32 i = (s & 3) * 8 + ((r - 32) >> 2); u0 = i & 3; r = 32 + ((i >> 2) << 2) + (r & 3); }
        } else {
            const bool a = order<KE + 32, 32 - KE, 5, NCHUNK - HEAD_STEPS / 4>(job - HEAD_JOBS, c, r);
            c += HEAD_STEPS / 4;
            r = !a ? KE + r : r < KE ? r : r - KE + 32;
        }
        if (r < 32) {
            const u32 t = r, s = c * 4 + u0;
            jsrc = (t * 32 + (lane >> 3)) * 16384 + s * 128 + (lane & 7) * 16; jstep = 128; jpiece = 131072;
            jslot = PK_X + (s * 32 + t) * 4096; jslot_step = 32 * 4096;
            jhdr = HDR_X + ((rank * 32 + t) * 128 + s) * 8;
            jkind = t < KEPT ? 4 : 5;
        } else {
            const u32 q = r & 3, e = (c * 8 + ((r - 32) >> 2)) * 4 + u0;
            jsrc = (e * 8 + (lane >> 5)) * 2048 + q * 512 + (lane & 31) * 16; jstep = 16384; jpiece = 4096;
            jslot = PK_W + (e * 4 + q) * 4096; jslot_step = 4 * 4096;
            jhdr = HDR_W + ((rank & 3) * 1024 + e) * 8;
            jkind = q;
        }
    };
    u32x4 nx[4];
    auto job_load = [&]() {
        const auto in = mkbuf(jkind >= 4 ? P.x : P.w, 16 * MiB);
#pragma unroll
        for (int j = 0; j < 4; ++j) nx[j] = __builtin_amdgcn_raw_buffer_load_b128(in, jsrc, j * jpiece, 0);
        jsrc += jstep;
    };
    const u32 ng = P.n_gemm, gw = wid * (NWG - ng) + ridx;
    if (cb != 0xffu || gw >= PER * 7) {
        const u32 nr = P.n_remote;
        job = cb == 0xffu ? HEAD_JOBS + (ng - nr) * 8 * AGK_LR + gw - PER * 7 : ridx < nr ? wid * nr + ridx : HEAD_JOBS + wid * (ng - nr) + (ridx - nr);
        if (cb != 0xffu && (wid == 8 || (ridx < nr && job >= HEAD_JOBS))) job = NJOBS;
        if (job < NJOBS) {
            if (job >= HEAD_JOBS) for (u32 i = 0; i < AGK_DELAY0 + (job - HEAD_JOBS) / AGK_SLOPE; ++i) __builtin_amdgcn_s_sleep(8);
            job_setup();
            job_load();
        }
    }
    const u32 call = uni(ctr_v) + 1u;
    if (lane == 0) __hip_atomic_store(ctr, call, __ATOMIC_RELAXED, __HIP_MEMORY_SCOPE_AGENT);
    const u32 par = call & 1u;
    const size_t hdr_par = OFF_HDR + par * HDR_PAR;

    if (cb == 0xffu && gw < PER * 7) {
        __builtin_amdgcn_s_setprio(AGK_PRIO);
        const u32 dir = gw % 7 + 1;
        const u32 p = (rank ^ dir) & 7u;
        pull_link(ws, P.out, P.ws[p] + OFF_PK + par * PK_PAR, dir, p, rank, gw / 7, (u32)hdr_par, call, lane,
                  lds + wid * CODEC_BYTES,
                  (u64*)(ws + OFF_DBG) + (blockIdx.x * 8 + wid) * 8);
        if (dir < 4) STAMP(1); else STAMP(2);
        return;
    }

    const auto pk = mkbuf(ws + OFF_PK + par * PK_PAR, PK_PAR);
    const u32 ctab = compact_table(lane);
    LAS char* const st = lds + LDS_CODEC + wid * CODEC_BYTES;
    while (job < NJOBS) {
        const u32 win = choose_window(nx[0]);
        u32 hd = 0;
        for (u32 u = 0; u < jn; ++u) {
            const u32x4 r[4] = {nx[0], nx[1], nx[2], nx[3]};
            if (u + 1 < jn) job_load();
            const u32 meta = pack_unit(r, win, ctab, st, pk, jslot + u * jslot_step, lane,
                                       [&]() { asm volatile("" : "+v"(nx[0]), "+v"(nx[1]), "+v"(nx[2]), "+v"(nx[3])); });
            if ((u32)(lane & 3) == u) hd = meta;
        }
        char* b = nullptr;
#pragma unroll
        for (int q = 0; q < 4; ++q) {
            char* pq = P.ws[(jkind == 4 ? rank ^ (q < 3 ? q + 1 : 1) : jkind == 5 ? oth | q : oth | jkind) & 7];
            if ((lane >> 2) == q) b = pq;
        }
        const bool on = (u32)(lane & 3) < jn && (u32)(lane >> 2) < (jkind == 4 ? 3u : jkind == 5 ? 4u : 1u);
        u64* const hp = (u64*)(b + hdr_par + jhdr) + (lane & 3);
        job = cb == 0xffu ? job + (NWG - ng) * 9 - PER * 7 : cb < 4 && job < HEAD_JOBS + (ng - P.n_remote) * 8 * (AGK_LR - 1) ? job + (ng - P.n_remote) * 8 : NJOBS;
        WAIT_VM(0);
        if (on) __hip_atomic_store(hp, ((u64)(call & TAG_MASK) << 32) | hd, __ATOMIC_RELAXED, __HIP_MEMORY_SCOPE_SYSTEM);
        if (job < NJOBS) { for (int i = 0; i < AGK_SPSLEEP; ++i) __builtin_amdgcn_s_sleep(64); job_setup(); job_load(); }
    }
    if (cb == 0xffu) { STAMP(7); return; }
    STAMP(1);

    const int wm = wid >> 2, wn = wid & 3, fr = lane & 15, fq = lane >> 4;
    const u32 ns = (u32)(tile >> 8) & 0xffu;
    auto strip_of = [&](u32 i) -> u32 { return (u32)(tile >> (16 + 8 * i)) & 0xffu; };
    if (wid == 8) {
        const u32 i = lane & 15, jj = lane >> 4;
        const u32 gs = strip_of(i < 5 ? i : 0);
        const u32 off = i < 5 ? OFF_XFLAG + (gs * 128 + jj) * 4 : OFF_WFLAG + (((cb & 3) * 128 + jj) * 8 + ((i - 5) & 7)) * 4;
        const u32 sh = i < 5 ? 2 : 5;
        const u64 skip = __builtin_amdgcn_ballot_w64(i < 5 ? (gs >> 5) == (u32)rank : (cb < 4 || i >= 13));
        const auto ctrl = mkbuf(ws, CTRL_BYTES);
        const u64 t_in = __builtin_amdgcn_s_memrealtime();
        u32 ready = skip == ~0ull ? NSTEP : 0;
        for (u32 s = 0; s < NSTEP; ++s) {
            if (AGK_TAU && cb < 4) while (__builtin_amdgcn_s_memrealtime() - t_in < AGK_T0 + s * AGK_TAU) __builtin_amdgcn_s_sleep(2);
            if (s >= ready) {
                const u64 t0 = __builtin_amdgcn_s_memrealtime();
                for (u32 it = 1;; ++it) {
                    asm volatile("" ::: "memory");
                    const u32 v = __builtin_amdgcn_raw_buffer_load_b32(ctrl, off + (s << sh), 0, 17);
                    const u64 m = ~(__builtin_amdgcn_ballot_w64(v == call) | skip);
                    ready = s + (m == 0ull ? 4u : (u32)__builtin_ctzll(m) >> 4);
                    if (s < ready) break;
                    if ((it & 63u) == 0u && give_up(tmo, call, t0)) { WAIT_VM(0); ready = NSTEP; break; }
                    __builtin_amdgcn_s_sleep(AGK_PSLEEP);
                }
            }
            asm volatile("" ::: "memory");
            __builtin_amdgcn_s_barrier();
            if (s == 2) __builtin_amdgcn_s_barrier();
        }
        return;
    }
    const u32 s01 = strip_of((wid & 2) + wm), h01 = wid & 1, s2 = strip_of(4);
    const bool own01 = (s01 >> 5) == (u32)rank, own2 = (s2 >> 5) == (u32)rank;
    const char* gA01 = own01 ? P.x + (size_t)(s01 & 31) * (32 * 16384) : ws + OFF_XRAW + (size_t)s01 * 4096;
    const char* gA2 = own2 ? P.x + (size_t)(s2 & 31) * (32 * 16384) : ws + OFF_XRAW + (size_t)s2 * 4096;
    const char* gB = cb < 4 ? P.w + cb * 512 : ws + OFF_WRAW + (cb & 3) * 32768;
    const u32 dA01 = own01 ? 128u : 1048576u, dA2 = own2 ? 128u : 1048576u;
    const u32 rs01 = own01 ? 16384u : 128u, rs2 = own2 ? 16384u : 128u, rsB = cb < 4 ? 2048u : 512u;
    const u32 l3 = lane >> 3;
    const u32 vA0 = (h01 * 16 + l3) * rs01 + (((lane & 7) ^ (l3 >> 1)) * 16);
    const u32 vA1 = (h01 * 16 + 8 + l3) * rs01 + (((lane & 7) ^ (4 + (l3 >> 1))) * 16);
    const u32 vA2 = (4 * wid + (l3 & 3)) * rs2 + (((lane & 7) ^ (((4 * wid + (l3 & 3)) >> 1) & 7)) * 16);
    u32 vB[2];
#pragma unroll
    for (int i = 0; i < 2; ++i) {
        const u32 k = 8 * wid + 2 * i + (lane >> 5), f = (k & 3) | (((k >> 3) & 1) << 2);
        vB[i] = k * rsB + (((lane & 31) ^ (f << 1)) * 16);
    }
    const u32 lds0 = (u32)(size_t)lds;
    const u32 rA0 = (wm * 80 + fr) * 128 + ((fq ^ (fr >> 1)) * 16), rA1 = rA0 ^ 64;
    u32 rB[4];
    {
        const u32 q = fr >> 2, p4 = fr & 3, f = q | ((fq & 1) << 2);
#pragma unroll
        for (int n = 0; n < 4; ++n) rB[n] = A_BYTES + (8 * fq + q) * 512 + (((wn * 4 + n) ^ f) * 32) + p4 * 8;
    }

    auto stage_piece = [&](int i, u32 slot_base) {
        if (i < 2) glds16(gA01, i ? vA1 : vA0, slot_base + (wid + wm) * 2048 + i * 1024);
        else if (i < 6) glds16(gB + (i >= 4 ? 4 * rsB : 0u), vB[i & 1], slot_base + A_BYTES + wid * 4096 + (i - 2) * 1024);
        else glds16_half(gA2, vA2, slot_base + 8192 + wm * 8192 + wid * 512);
    };
    auto advance = [&]() { gA01 += dA01; gA2 += dA2; gB += 131072; };

    f32x4 acc[5][4];
#pragma unroll
    for (int m = 0; m < 5; ++m)
#pragma unroll
        for (int n = 0; n < 4; ++n) acc[m][n] = (f32x4){0.f, 0.f, 0.f, 0.f};
    s16x8 fa[2][5];
    s16x4 fb[2][4][2];
    auto frag_read = [&](int set, int i, u32 sb, int kk) {
        if (i == 2) fa[set][0] = *(const LAS s16x8*)(lds + sb + (kk ? rA1 : rA0));
        else if (i >= 9) fa[set][i - 8] = *(const LAS s16x8*)(lds + sb + (kk ? rA1 : rA0) + (i - 8) * 2048);
        else {
            const int b = i < 2 ? i : i - 1, n = b >> 1, t = b & 1;
            fb[set][n][t] = __builtin_amdgcn_ds_read_tr16_b64_v4i16((LAS s16x4*)(lds + sb + rB[n] + t * 2048 + kk * 16384));
        }
    };
    auto mma = [&](int set, int i) {
        const int m = i >> 2, n = i & 3;
        const s16x8 b = __builtin_shufflevector(fb[set][n][0], fb[set][n][1], 0, 1, 2, 3, 4, 5, 6, 7);
        acc[m][n] = __builtin_amdgcn_mfma_f32_16x16x32_bf16(__builtin_bit_cast(bf16x8, b), __builtin_bit_cast(bf16x8, fa[set][m]), acc[m][n], 0, 0, 0);
    };

#define STAMP_CLK() do {} while (0)
    u32 cur = 0, tmo_v;
#define KSTEP(MB, LOAD, VMW, NEXT) do { \
        const u32 nxt = cur == 2 * STAGE ? 0u : cur + STAGE; \
        _Pragma("unroll") for (int i = 0; i < 4 * MB; ++i) { mma(0, i); SCHED(); if (i < 8 + MB) frag_read(1, i, cur, 1); SCHED(); } \
        if (LOAD || t == 0) WAIT_VM(VMW); else WAIT_VM(0); \
        WAIT_LGKM0(); \
        if (NEXT) { __builtin_amdgcn_s_barrier(); asm volatile("" ::: "memory"); } \
        SCHED(); \
        _Pragma("unroll") for (int i = 0; i < 4 * MB; ++i) { \
            mma(1, i); SCHED(); \
            if (NEXT && i < 8 + MB) frag_read(0, i, nxt, 0); \
            if (LOAD && i >= 3 * MB - 2) stage_piece(i - (3 * MB - 2), lds0 + cur); \
            SCHED(); } \
        if (LOAD) advance(); \
        cur = nxt; } while (0)
#define KLOOP(MB, NG, NG2) do { \
        __builtin_amdgcn_s_waitcnt(0x0F70);     \
        _Pragma("unroll") for (int s = 0; s < 3; ++s) { \
            __builtin_amdgcn_s_barrier(); \
            if (s == 0) STAMP(2); \
            _Pragma("unroll") for (int i = 0; i < NG; ++i) stage_piece(i, lds0 + s * STAGE); \
            advance(); } \
        WAIT_VM(NG2); \
        __builtin_amdgcn_s_barrier(); \
        asm volatile("" ::: "memory"); \
        _Pragma("unroll") for (int i = 0; i < 8 + MB; ++i) frag_read(0, i, 0, 0); \
        STAMP_CLK(); \
        for (u32 t = 0; t < NSTEP - 3; ++t) KSTEP(MB, true, NG, true); \
        tmo_v = __hip_atomic_load(tmo, __ATOMIC_RELAXED, __HIP_MEMORY_SCOPE_AGENT);     \
        _Pragma("nounroll") for (u32 t = 0; t < 3; ++t) KSTEP(MB, false, NG, true); } while (0)
    if (ns > 4) KLOOP(5, 7, 14); else KLOOP(4, 6, 12);
#undef KLOOP
#undef KSTEP
    STAMP(3);

    const u32 nanm = tmo_v == call ? 0x7fc07fc0u : 0u;
    int tz = threadIdx.x;
    asm volatile("" : "+v"(tz));
    const int ln = tz & 63;
    LAS char* const img = lds + (wm * 32 + (ln & 15)) * EP_STRIDE + (wn * 64 + (ln >> 4) * 4) * 2;
    auto act = [&](int m) {
        LAS char* q = img + (m < 4 ? (m >> 1) * 64 + (m & 1) * 16 : 128 - wm * 16) * EP_STRIDE;
#pragma unroll
        for (int n = 0; n < 4; ++n) {
            const f32x4 a = acc[m][n];
            *(LAS u32x2*)(q + n * 32) = (u32x2){silu2(a[0], a[1]) | nanm, silu2(a[2], a[3]) | nanm};
        }
    };
    const auto out = mkbuf(P.out, 16 * MiB);
    const u32 q4 = cb & 3, ctab2 = compact_table(ln);
    u64* const hbase = (u64*)(P.ws[(oth | q4) & 7] + hdr_par + HDR_O) + (rank & 3) * 1024;
    u32 win = 0;
    asm volatile("" ::: "memory");
    act(0);
    auto phase = [&](auto PH) {
        constexpr u32 ph = decltype(PH)::value;
        if (ph == 0) act(1);
        if (ph == 1) act(3);
        WAIT_LGKM0();
        __builtin_amdgcn_s_barrier();
        asm volatile("" ::: "memory");
        const u32 u = ph * 8 + wid, e = strip_of(u >> 2) * 4 + (u & 3);
        const bool on = u < ns * 4;
        u32 meta = 0;
        if (on) {
            u32x4 r[4];
#pragma unroll
            for (int j = 0; j < 4; ++j) r[j] = *(const LAS u32x4*)(lds + (u * 8 + 2 * j + (ln >> 5)) * EP_STRIDE + (ln & 31) * 16);
            if (cb < 4) {
#pragma unroll
                for (int j = 0; j < 4; ++j)
                    __builtin_amdgcn_raw_buffer_store_b128(r[j], out, (ln >> 5) * 2048 + (ln & 31) * 16 + e * 16384 + j * 4096 + cb * 512, 0, 16);
            } else {
                if (ph == 0) win = choose_window(r[0]);
                meta = pack_unit(r, win, ctab2, st, pk, PK_O + (q4 * 1024 + e) * 4096, ln, []() {});
            }
        }
        asm volatile("" ::: "memory");
        u32 seq = __builtin_amdgcn_raw_ptr_buffer_atomic_add_i32(1, mkbuf(ws, CTRL_BYTES), ln == 0 && on && cb >= 4 ? (u32)OFF_OSEQ + q4 * 128 : 0xfffffff0u, 0, 0);
        asm volatile("" ::: "memory");
        SCHED();
        if (ph == 0) act(2);
        if (ph == 1 && ns > 4) act(4);
        WAIT_VM(0);
        asm volatile("" : "+v"(seq));
        if (on && cb >= 4) {
            if (ln == 0) __hip_atomic_store(hbase + ((seq - (call - 1u) * U_OUT) & 1023u), ((u64)((call & TAG_MASK) | (e << 22)) << 32) | meta,
                                            __ATOMIC_RELAXED, __HIP_MEMORY_SCOPE_SYSTEM);
        }
        SCHED();
    };
    phase(Const<0>());
    phase(Const<1>());
    if (ns > 4) phase(Const<2>());
    STAMP(4);
}

static void build_tiles(int rank, u64* tile, int* n_gemm = nullptr, int* n_remote = nullptr) {
    const int oth = (rank & 4) ^ 4;
    u64 rows[8][40];
    int key[8][40], nrow[8];
    for (int cb = 0; cb < 8; ++cb) {
        int q[160], pos[160], size[160], nq = 0;
        auto add = [&](int gs, int p, int sz) { q[nq] = gs; pos[nq] = p; size[nq++] = cb < 4 ? 5 : sz; };
        if (cb < 4) for (int t = KEPT; t < 32; ++t) add(rank * 32 + t, 0, 5);
        for (int t = 0; t < KEPT; ++t) add(rank * 32 + t, 20 + t, 5);
        for (int d = 1; d < 4; ++d) for (int t = 0; t < KA; ++t) add((rank ^ d) * 32 + t, 100 + d * 32 + t, 5);
        for (int i = 0; i < 4; ++i) if (cb < 4 || (cb & 3) != i) for (int t = KEPT; t < 32; ++t) add((oth | i) * 32 + t, 300 + i * 32 + t, (AGK_SMALL & 1) ? 4 : 5);
        for (int d = 1; d < 4; ++d) for (int t = KA; t < KA + KM; ++t) add((rank ^ d) * 32 + t, 500 + d * 32 + t, (AGK_SMALL & 2) ? 4 : 5);
        for (int d = 1; d < 4; ++d) for (int t = KA + KM; t < KEPT; ++t) add((rank ^ d) * 32 + t, 700 + d * 32 + t, (AGK_SMALL & 4) ? 4 : 5);
        nrow[cb] = 0;
        while (nq > 0) {
            int left = 0;
            auto grp = [&](int i) { return cb < 4 ? 0 : pos[i] < 300 ? 0 : pos[i] < 500 ? 1 : 2; };
            while (left < nq && grp(nq - 1 - left) == grp(nq - 1)) ++left;
            const int nt = (left + size[nq - 1] - 1) / size[nq - 1], n = (left + nt - 1) / nt, s0 = nq - n;
            u64 e = (u64)cb | ((u64)n << 8);
            for (int s = 0; s < 5; ++s) e |= (u64)q[nq - 1 - (s < n ? s : 0)] << (16 + 8 * s);
            key[cb][nrow[cb]] = pos[s0 + n / 2];
            rows[cb][nrow[cb]++] = e;
            nq = s0;
        }
    }
    u64 list[8][32];
    int nl[8];
    for (int half = 0; half < 2; ++half) {
        int total = 0, n = 0;
        for (int c = 0; c < 4; ++c) { total += nrow[AGK_PURE ? half * 4 + c : (c >> 1) * 4 + half * 2 + (c & 1)]; nl[half * 4 + c] = 0; }
        for (int k = 0; k < 900; ++k)
            for (int c = 0; c < 4; ++c) {
                const int cb = AGK_PURE ? half * 4 + c : (c >> 1) * 4 + half * 2 + (c & 1);
                for (int r = 0; r < nrow[cb]; ++r)
                    if (key[cb][r] == k) { const int x = half * 4 + n * 4 / total; list[x][nl[x]++ & 31] = rows[cb][r]; ++n; }
            }
    }
    int ng = 0, nc = 0;
    for (int b = 0; b < NWG; ++b) {
        const int xcd = AGK_PURE == 2 ? (b & 1) * 4 + ((b & 7) >> 1) : b & 7, i = (b >> 3) - (32 - nl[xcd]);
        tile[b] = i < 0 ? 0xffull | ((u64)nc++ << 56) : list[xcd][i];
    }
    for (int cls = 0; cls < 2; ++cls) {
        for (int b = 0; b < NWG; ++b) {
            const int cb = (int)(tile[b] & 0xff);
            if (cb != 0xff && (cb < 4) == (cls == 1)) tile[b] |= (u64)ng++ << 56;
        }
        if (cls == 0 && n_remote) *n_remote = ng;
    }
    if (n_gemm) *n_gemm = ng;
}
}

extern "C" void kernel_launch(int rank, int nranks, ncclComm_t comm,
                              hipStream_t stream,
                              void* const* d_in, const int* in_sizes, int n_in,
                              void* d_out, int out_size,
                              const int* in_dtypes, int out_dtype,
                              void* const* d_out_all,
                              void* const* d_ws_all, size_t ws_bytes) {
    using namespace agk;
    if (nranks != NRANK || (unsigned)rank >= (unsigned)NRANK || ws_bytes < WS_NEED || n_in < 2 ||
        in_sizes[0] != 1024 * KDIM || in_sizes[1] != KDIM * 1024 || out_size != 8192 * 1024) {
        fprintf(stderr, "kernel_launch: unexpected problem shape\n");
        return;
    }
    Params P;
    for (int p = 0; p < NRANK; ++p) P.ws[p] = (char*)d_ws_all[p];
    P.x = (const char*)d_in[0];
    P.w = (const char*)d_in[1];
    P.out = (char*)d_out;
    P.rank = rank;
    P.self = P.ws[rank];
    P.pad = 0;
    build_tiles(rank, P.tile, &P.n_gemm, &P.n_remote);
    if ((u32)P.n_remote * 8 < HEAD_JOBS || (NWG - P.n_gemm) * 9 <= PER * 7) return;

    hipLaunchKernelGGL(ag_gemm_silu, dim3(NWG), dim3(576), 0, stream, P);
    HIP_CHECK(hipGetLastError());
}
```

```cpp
#include <hip/hip_runtime.h>
#include <rccl/rccl.h>
#include <cstdio>

#define HIP_CHECK(expr) do { hipError_t _e = (expr); if (_e != hipSuccess) { \
    fprintf(stderr, "HIP error %s at %s:%d\n", hipGetErrorString(_e), __FILE__, __LINE__); \
    return; } } while (0)

namespace agk {
#define LAS __attribute__((address_space(3)))
typedef unsigned u32;
typedef unsigned long long u64;
typedef unsigned u32x4 __attribute__((ext_vector_type(4)));
typedef unsigned u32x2 __attribute__((ext_vector_type(2)));
typedef short s16x4 __attribute__((ext_vector_type(4)));
typedef short s16x8 __attribute__((ext_vector_type(8)));
typedef __bf16 bf16x8 __attribute__((ext_vector_type(8)));
typedef __bf16 bf16x2 __attribute__((ext_vector_type(2)));
typedef float f32x4 __attribute__((ext_vector_type(4)));
typedef float f32x2 __attribute__((ext_vector_type(2)));

constexpr int NRANK = 8, KDIM = 8192;
constexpr int F = 12, KEPT = 32 - F;
constexpr int NWG = 256, NLOC = 112, NREM = 96, NCOMM = NWG - NLOC - NREM;
constexpr int PER = 27;
constexpr int KC = 5, LEAD0 = 18, LEAD1 = 14, LEAD2 = 11;
constexpr int LEAD_W = 22;
constexpr u32 EVEN_STEPS = 8;
static_assert(KEPT == 4 * KC, "four classes");
constexpr u32 PACE_T0 = 2000, PACE_STEP = 91;
constexpr int NCHUNK = 32;
constexpr int NSTEP = KDIM / 64;
constexpr u32 HEAD_STEPS = 12, HEAD_JOBS = HEAD_STEPS * 64, NJOBS = HEAD_JOBS + 2048 - HEAD_STEPS * 16;
constexpr u32 NLATE = 44;
constexpr u32 ROUND2 = 4;
constexpr u32 PUSH_STEPS = 1;
static_assert(PUSH_STEPS <= 3 && PUSH_STEPS <= HEAD_STEPS && PUSH_STEPS <= EVEN_STEPS, "handled by the K loop prologue");
static_assert(PUSH_STEPS * 64 + NREM * 8 >= HEAD_JOBS && PER * 7 <= NCOMM * 4 && PUSH_STEPS * 7 * 21 <= NLOC * 8, "enough waves for every job");
constexpr u64 DEADLINE = 50000000ull;

constexpr size_t MiB = 1ull << 20;
constexpr size_t OFF_CTR = 0;
constexpr size_t OFF_TMO = 16384;
constexpr size_t OFF_TICKET = 32 * 1024;
constexpr size_t OFF_OSEQ = 40 * 1024;
constexpr u32 V_W = 4096, V_O = 5120;
constexpr size_t OFF_FLAG = 256 * 1024;
constexpr size_t OFF_HDR = 1 * MiB;
constexpr size_t HDR_SRC = 192 * 1024, HDR_PAR = 8 * HDR_SRC;
constexpr u32 HDR_OUT = 64 * 1024;
constexpr size_t CTRL_BYTES = 4 * MiB;
constexpr size_t OFF_XRAW = CTRL_BYTES;
constexpr size_t OFF_WRAW = OFF_XRAW + 132 * MiB;
constexpr u32 XS = 132 * 131072, WS = 132 * 32768;
constexpr size_t OFF_PK = OFF_WRAW + 18 * MiB;
constexpr size_t PK_PAR = 48 * MiB;
constexpr u32 Z_W = 4096, Z_O = 8192;
constexpr size_t OFF_SIDE = OFF_PK + 2 * PK_PAR;
constexpr size_t SIDE_SRC = V_O / 4 * 256, WS_NEED = OFF_SIDE + 16 * SIDE_SRC;

constexpr int A_BYTES = 160 * 128, STAGE = A_BYTES + 64 * 512, LDS_BYTES = 3 * STAGE;
constexpr int LDS_TABLE = 32 * 1024;
constexpr int LDS_LOCK = 64 * 1024;
constexpr int LDS_CODEC = 96 * 1024, CODEC_BYTES = 3072, UNPACK_BYTES = 4096;

template <u32 V> struct Const { static constexpr u32 value = V; };

struct Params {
    char* ws[NRANK];
    const char* x;
    const char* w;
    char* out;
    char* self;
    int rank, pad;
    u64 tile[NWG];
};

#define WAIT_VM(n) asm volatile("s_waitcnt vmcnt(" #n ")" ::: "memory")
#define WAIT_LGKM0() do { asm volatile("" ::: "memory"); __builtin_amdgcn_s_waitcnt(0xC07F); asm volatile("" ::: "memory"); } while (0)
#define SCHED() __builtin_amdgcn_sched_barrier(0)

__device__ __forceinline__ __amdgpu_buffer_rsrc_t mkbuf(const void* p, u32 bytes) {
    return __builtin_amdgcn_make_buffer_rsrc((void*)p, 0, bytes, 0x00020000);
}
__device__ __forceinline__ u32 uni(u32 v) { return __builtin_amdgcn_readfirstlane(v); }

__device__ __forceinline__ bool give_up(u32* tmo, u32 call, u64 t0) {
    if (uni(__hip_atomic_load(tmo, __ATOMIC_RELAXED, __HIP_MEMORY_SCOPE_AGENT)) == call ||
        __builtin_amdgcn_s_memrealtime() - t0 > DEADLINE) {
        __hip_atomic_store(tmo, call, __ATOMIC_RELAXED, __HIP_MEMORY_SCOPE_AGENT);
        return true;
    }
    return false;
}

typedef unsigned short u16x2 __attribute__((ext_vector_type(2)));
__device__ __forceinline__ u32 pk_add(u32 a, u32 b) { return __builtin_bit_cast(u32, __builtin_bit_cast(u16x2, a) + __builtin_bit_cast(u16x2, b)); }
__device__ __forceinline__ u32 pk_sub(u32 a, u32 b) { return __builtin_bit_cast(u32, __builtin_bit_cast(u16x2, a) - __builtin_bit_cast(u16x2, b)); }
__device__ __forceinline__ u32 pk_min(u32 a, u32 b) { return __builtin_bit_cast(u32, __builtin_elementwise_min(__builtin_bit_cast(u16x2, a), __builtin_bit_cast(u16x2, b))); }
__device__ __forceinline__ u32 perm(u32 hi, u32 lo, u32 sel) { return __builtin_amdgcn_perm(hi, lo, sel); }
__device__ __forceinline__ u32 wave_scan(u32 v) {
    v += (u32)__builtin_amdgcn_update_dpp(0, (int)v, 0x111, 0xf, 0xf, true);
    v += (u32)__builtin_amdgcn_update_dpp(0, (int)v, 0x112, 0xf, 0xf, true);
    v += (u32)__builtin_amdgcn_update_dpp(0, (int)v, 0x114, 0xf, 0xf, true);
    v += (u32)__builtin_amdgcn_update_dpp(0, (int)v, 0x118, 0xf, 0xf, true);
    v += (u32)__builtin_amdgcn_update_dpp(0, (int)v, 0x142, 0xa, 0xf, false);
    v += (u32)__builtin_amdgcn_update_dpp(0, (int)v, 0x143, 0xc, 0xf, false);
    return v;
}
__device__ __forceinline__ u32 sym_to_d(u32 s, bool three) { return perm(three ? 0xf704fdfcu : 0xfffefdfcu, three ? 0xfbfaf9f8u : 0xfbfa0403u, s); }
__device__ __forceinline__ u32 compact_table(int lane) {
    u32 t = 0x0c0c0c0cu;
#pragma unroll
    for (int b = 3; b >= 0; --b) if ((lane >> b) & 1) t = (t << 8) | (u32)b;
    return t;
}
__device__ __forceinline__ u32 expand_table(int lane) {
    u32 t = 0, n = 0;
#pragma unroll
    for (int b = 0; b < 4; ++b) { const u32 on = (lane >> b) & 1; t |= (on ? n : 4u + b) << (8 * b); n += on; }
    return t;
}
__device__ __forceinline__ u32 esc_flags(const u32* E, int k) { return E[k >> 1] & ((k & 1) ? 0x04040404u : 0x01010101u); }
__device__ __forceinline__ u32 esc_lookup(u32 f, int k, u32 tab) {
    return (u32)__builtin_amdgcn_ds_bpermute((int)__builtin_amdgcn_udot4(f, (k & 1) ? 0x08040201u : 0x20100804u, 0u, false), (int)tab);
}
__device__ __forceinline__ u32 choose_window(u32x4 v) {
    u32 sum = 0;
#pragma unroll
    for (int i = 0; i < 4; ++i) sum = __builtin_amdgcn_udot2(__builtin_bit_cast(u16x2, v[i] & 0x7fff7fffu), (u16x2){1, 1}, sum, false);
    const u32 mean = (u32)__builtin_amdgcn_readlane(wave_scan(sum), 63) >> 9;
    return mean > 139u ? (mean - 139u) >> 3 : 0u;
}

__device__ __forceinline__ u32 out_unit(u32 e) { return (e >> 2) * 65536 + (e & 3) * 128; }
__device__ __forceinline__ u32 out_piece(int j) { return (j >> 1) * 32768 + (j & 1) * 64; }
__device__ __forceinline__ u32 out_lane(int lane) { return (lane & 15) * 2048 + (lane & 16) * 2 + (lane & 32) / 2; }
__device__ __forceinline__ u32 inline_words(u32 meta) {
    const u32 n = ((meta >> 15) & 0xffu) + ((meta >> 23) & 0x7fu) - 65u;
    return n < 15u ? n + 1u : 0u;
}

template <int N> __device__ __forceinline__ void scatter_bytes(const u32* D, const u32* f, u32 p) {
    asm volatile("" ::: "memory");
#pragma unroll
    for (int k = 0; k < N; ++k) {
        LAS u32* q = (LAS u32*)(size_t)(p & ~3u);
        __hip_atomic_fetch_or(q, __builtin_amdgcn_alignbyte(D[k], 0u, ~p), __ATOMIC_RELAXED, __HIP_MEMORY_SCOPE_WAVEFRONT);
        __hip_atomic_fetch_or(q + 1, __builtin_amdgcn_alignbyte(0u, D[k], ~p), __ATOMIC_RELAXED, __HIP_MEMORY_SCOPE_WAVEFRONT);
        p += __builtin_popcount(f[k]);
    }
    asm volatile("" ::: "memory");
}

template <class Pre>
__device__ __forceinline__ u32 pack_unit(const u32x4 (&r)[4], u32 win, bool three, u32 ctab, LAS char* st, __amdgpu_buffer_rsrc_t dst,
                                         u32 slot, int lane, u32& t0, u32& t1, u32& nt, Pre before_stores) {
    constexpr u32 S1 = 16, S2 = 1552;
    const u32 B2 = ((win & 0xfffu) << 4) * 0x10001u;
    u32 L[8], D[8], C[2] = {0u, 0u};
#pragma unroll
    for (int k = 0; k < 8; ++k) {
        const u32 d0 = r[k >> 1][(k & 1) * 2], d1 = r[k >> 1][(k & 1) * 2 + 1];
        const u32 x0 = pk_sub(__builtin_amdgcn_alignbit(d0, d0, 31), B2), x1 = pk_sub(__builtin_amdgcn_alignbit(d1, d1, 31), B2);
        L[k] = perm(x1, x0, 0x06040200u);
        D[k] = perm(x1, x0, 0x07050301u);
        C[k >> 2] |= perm(pk_min(x1, 0x03ff03ffu), pk_min(x0, 0x03ff03ffu), 0x07050301u) << (2 * (k & 3));
    }
    const u32 E0 = C[0] & (C[0] >> 1) & 0x55555555u, E1 = C[1] & (C[1] >> 1) & 0x55555555u;
    const u32 E[4] = {E0, E0 >> 4, E1, E1 >> 4};
    *(LAS u32x4*)(st + lane * 16) = (u32x4){0u, 0u, 0u, 0u};
    *(LAS u32x4*)(st + 1024 + lane * 16) = (u32x4){0u, 0u, 0u, 0u};
    u32 f[8], sel[8];
#pragma unroll
    for (int k = 0; k < 8; ++k) {
        f[k] = esc_flags(E, k);
        sel[k] = esc_lookup(f[k], k, ctab);
    }
    SCHED();
    const u32 tot = __builtin_popcount(E0) + __builtin_popcount(E1);
    const u32 incl = wave_scan(tot), n = __builtin_amdgcn_readlane(incl, 63);
#pragma unroll
    for (int k = 0; k < 8; ++k) { D[k] = perm(0u, D[k], sel[k]); asm volatile("" : "+v"(D[k])); }
    u32 nsw = 0, np = 0;
    t0 = t1 = nt = 0;
    bool raw = n > 1280u;
    if (!raw) {
        scatter_bytes<8>(D, f, incl - tot + (u32)(size_t)st + S1 - 1u);
        *(LAS unsigned char*)(st + S1 + n + (lane & 15)) = 0xff;
        asm volatile("" ::: "memory");
        u32 w2 = 0, n2w = 0, n3 = 0;
        three = three && n <= 960u;
        if (three) {
            const u32x4 v = *(const LAS u32x4*)(st + S1 + lane * 16);
            u32 d[4], g[4], sl[4];
#pragma unroll
            for (int k = 0; k < 4; ++k) {
                const u32 i7 = v[k] & 0x07070707u;
                const u32 other = (__builtin_amdgcn_lerp(perm(0xfffe0000u, 0x03000000u, i7) ^ v[k], 0xffffffffu, 0u) >> 7) & 0x01010101u;
                w2 |= (perm(0x00010303u, 0x02030303u, i7) | other | (other << 1)) << (2 * k);
            }
            n2w = (n + 15u) >> 4;
            if ((u32)lane >= n2w) w2 = 0u;
            const u32 G0 = w2 & (w2 >> 1) & 0x55555555u, G[2] = {G0, G0 >> 4};
#pragma unroll
            for (int k = 0; k < 4; ++k) { g[k] = esc_flags(G, k); sl[k] = esc_lookup(g[k], k, ctab); }
            const u32 tot3 = __builtin_popcount(G0), incl3 = wave_scan(tot3);
            n3 = __builtin_amdgcn_readlane(incl3, 63);
#pragma unroll
            for (int k = 0; k < 4; ++k) d[k] = perm(0u, v[k], sl[k]);
            three = n3 <= 470u;
            if (three) {
                if (tot3 != 0u) scatter_bytes<4>(d, g, incl3 - tot3 + (u32)(size_t)st + S2 - 1u);
                *(LAS unsigned char*)(st + S2 + n3 + (lane & 15)) = 0xfd;
                asm volatile("" ::: "memory");
            }
        }
        LAS u32* const list = (LAS u32*)(st + 2048);
        for (;;) {
            nsw = ((three ? n3 : n) + 9u) / 10u;
            np = 0;
            for (u32 i0 = 0; i0 < nsw; i0 += 64) {
                const u32 i = i0 + lane;
                const LAS u32* q = (const LAS u32*)(st + (three ? S2 : S1) + ((i * 10u) & ~3u));
                const u32 a0 = q[0], a1 = q[1], a2 = q[2], sh = (lane & 1) * 2;
                const u32 d[3] = {__builtin_amdgcn_alignbyte(a1, a0, sh), __builtin_amdgcn_alignbyte(a2, a1, sh), __builtin_amdgcn_alignbyte(0u, a2, sh)};
                u32 sy[3], x[3];
#pragma unroll
                for (int k = 0; k < 3; ++k) {
                    sy[k] = (d[k] ^ perm(three ? 2u : 5u, 0x03000000u, (d[k] | (d[k] >> 4)) & 0x07070707u)) & 0x07070707u;
                    x[k] = sym_to_d(sy[k], three) ^ d[k];
                }
                x[2] &= 0xffffu;
                if (i >= nsw) x[0] = x[1] = x[2] = 0u;
                const u32 w = sy[0] | (sy[1] << 3) | (((sy[2] & 0x0303u) | ((sy[2] & 0x0404u) << 14)) << 6);
                if (i0 == 0) t0 = w; else t1 = w;
                for (;;) {
                    const bool has = (x[0] | x[1] | x[2]) != 0u;
                    const u64 m = __builtin_amdgcn_ballot_w64(has);
                    if (m == 0ull || np > 127u) break;
                    const u32 ws = x[0] ? 0u : x[1] ? 1u : 2u;
                    const u32 xw = x[0] ? x[0] : x[1] ? x[1] : x[2], dw = x[0] ? d[0] : x[1] ? d[1] : d[2];
                    const u32 b8 = (u32)__builtin_ctz(xw | 0x80000000u) & 24u;
                    const u32 rk = np + __builtin_amdgcn_mbcnt_hi((u32)(m >> 32), __builtin_amdgcn_mbcnt_lo((u32)m, 0u));
                    list[has ? rk : 255u] = ((i * 10u + ws * 4u + (b8 >> 3)) << 8) | ((dw >> b8) & 0xffu);
                    const u32 keep = ~(0xffu << b8);
                    x[0] &= ws == 0u ? keep : ~0u; x[1] &= ws == 1u ? keep : ~0u; x[2] &= ws == 2u ? keep : ~0u;
                    np += __builtin_popcountll(m);
                }
            }
            if (!three || nsw + np + n2w <= 64u) break;
            three = false;
        }
        if (three) list[np + lane] = w2;
        asm volatile("" ::: "memory");
        nt = nsw + np + (three ? n2w : 0u);
        raw = np > 127u || nt > 128u;
        const u32 l0 = list[(lane - nsw) & 127u], l1 = list[(lane + 64u - nsw) & 127u];
        if ((u32)lane >= nsw) t0 = l0;
        if (lane + 64u >= nsw) t1 = l1;
        asm volatile("" ::: "memory");
    }
    before_stores();
    if (!raw) {
        __builtin_amdgcn_raw_buffer_store_b128((u32x4){L[0], L[1], L[2], L[3]}, dst, slot + lane * 16, 0, 17);
        __builtin_amdgcn_raw_buffer_store_b128((u32x4){L[4], L[5], L[6], L[7]}, dst, slot + 1024 + lane * 16, 0, 17);
        __builtin_amdgcn_raw_buffer_store_b64((u32x2){C[0], C[1]}, dst, slot + 2048 + lane * 8, 0, 17);
        __builtin_amdgcn_raw_buffer_store_b32(t0, dst, slot + 2560 + lane * 4, 0, 17);
        if (nt > 64u) __builtin_amdgcn_raw_buffer_store_b32(t1, dst, slot + 2816 + lane * 4, 0, 17);
        return win | (three ? 0x1000u : 0u) | (nsw << 15) | (np << 23);
    }
#pragma unroll
    for (int j = 0; j < 4; ++j) __builtin_amdgcn_raw_buffer_store_b128(r[j], dst, slot + j * 1024 + lane * 16, 0, 17);
    return 0x80000000u;
}

template <bool three>
__device__ __forceinline__ void unpack_tail(u32 (&f)[8], u32 (&sel)[8], u32& p0, u32x2 C, u32 t0, u32 t1, u32 meta, u32 xtab, LAS char* st, int lane) {
    const u32 nsw = (meta >> 15) & 0xffu, np = (meta >> 23) & 0x7fu;
    LAS char* const sy = st + (three ? 2560 : 0);
    auto expand = [&](u32 w, u32 i) {
        const u32 t = (w >> 6) & 0x03030303u;
        const u32 d0 = sym_to_d(w & 0x07070707u, three);
        const u32 d1 = sym_to_d((w >> 3) & 0x07070707u, three);
        const u32 d2 = sym_to_d(t | (t >> 14), three);
        volatile LAS unsigned short* q = (volatile LAS unsigned short*)(sy + i * 10u);
        q[0] = (unsigned short)d0; q[1] = (unsigned short)(d0 >> 16); q[2] = (unsigned short)d1; q[3] = (unsigned short)(d1 >> 16);
        q[4] = (unsigned short)d2;
    };
    auto patch = [&](u32 w, u32 i) {
        *(volatile LAS unsigned char*)(sy + (i - nsw < np ? min(w >> 8, 1400u) : 1400u)) = (unsigned char)w;
    };
    expand(t0, lane);
    if (!three && nsw + np > 64u) { expand(t1, lane + 64u); patch(t1, lane + 64u); }
    if (!three || np != 0u) patch(t0, lane);
    asm volatile("" ::: "memory");
    const u32 E0 = C[0] & (C[0] >> 1) & 0x55555555u, E1 = C[1] & (C[1] >> 1) & 0x55555555u;
    const u32 E[4] = {E0, E0 >> 4, E1, E1 >> 4};
    const u32 w2 = three && (u32)lane >= nsw + np ? t0 : 0u, G0 = w2 & (w2 >> 1) & 0x55555555u;
    const u32 tot = __builtin_popcount(E0) + __builtin_popcount(E1) + (__builtin_popcount(G0) << 16);
    const u32 G[2] = {G0, G0 >> 4};
    u32 g[4], sl[4];
#pragma unroll
    for (int k = 0; k < (three ? 4 : 8); ++k) {
        if (three) { g[k] = esc_flags(G, k); sl[k] = esc_lookup(g[k], k, xtab); }
        else { f[k] = esc_flags(E, k); sel[k] = esc_lookup(f[k], k, xtab); }
    }
    const u32 excl = wave_scan(tot) - tot;
    if (three) {
        u32 q3[5], a0[4], a1[4];
        q3[0] = (excl >> 16) + (u32)(size_t)sy;
#pragma unroll
        for (int k = 0; k < 4; ++k) {
            const LAS u32* q = (const LAS u32*)(size_t)(q3[k] & ~3u);
            a0[k] = q[0]; a1[k] = q[1];
            q3[k + 1] = q3[k] + __builtin_popcount(g[k]);
        }
        SCHED();
#pragma unroll
        for (int k = 0; k < 8; ++k) { f[k] = esc_flags(E, k); sel[k] = esc_lookup(f[k], k, xtab); }
        SCHED();
        u32x4 v;
#pragma unroll
        for (int k = 0; k < 4; ++k)
            v[k] = perm(perm(0u, 0x0003feffu, (w2 >> (2 * k)) & 0x03030303u), __builtin_amdgcn_alignbyte(a1[k], a0[k], q3[k]), sl[k]);
        *(LAS u32x4*)(st + ((lane - nsw - np) & 127u) * 16u) = v;
        asm volatile("" ::: "memory");
    }
    p0 = (three ? excl & 0xffffu : excl) + (u32)(size_t)st;
}
__device__ __forceinline__ void unpack_unit(u32x4 (&o)[4], u32x4 la, u32x4 lb, u32x2 C, u32 t0, u32 t1, u32 meta, u32 xtab, LAS char* st, int lane) {
    const u32 B2 = ((meta & 0xfffu) << 4) * 0x10001u;
    const u32 L[8] = {la[0], la[1], la[2], la[3], lb[0], lb[1], lb[2], lb[3]};
    u32 f[8], s0[8], s1[8], sel[8], p[9];
    if (meta & 0x1000u) unpack_tail<true>(f, sel, p[0], C, t0, t1, meta, xtab, st, lane);
    else unpack_tail<false>(f, sel, p[0], C, t0, t1, meta, xtab, st, lane);
#pragma unroll
    for (int k = 0; k < 8; ++k) {
        const LAS u32* q = (const LAS u32*)(size_t)(p[k] & ~3u);
        s0[k] = q[0]; s1[k] = q[1];
        p[k + 1] = p[k] + __builtin_popcount(f[k]);
    }
    SCHED();
#pragma unroll
    for (int k = 0; k < 8; ++k) {
        const u32 c = (C[k >> 2] >> (2 * (k & 3))) & 0x03030303u;
        const u32 d = perm(c, __builtin_amdgcn_alignbyte(s1[k], s0[k], p[k]), sel[k]);
        const u32 x0 = pk_add(perm(d, L[k], 0x05010400u), B2), x1 = pk_add(perm(d, L[k], 0x07030602u), B2);
        o[k >> 1][(k & 1) * 2] = __builtin_amdgcn_alignbit(x0, x0, 1);
        o[k >> 1][(k & 1) * 2 + 1] = __builtin_amdgcn_alignbit(x1, x1, 1);
    }
    asm volatile("" ::: "memory");
}

template <int A, int B, int L, int N>
__device__ __forceinline__ bool order(u32 n, u32& s, u32& r) {
    if (n < A * L) { s = n / A; r = n % A; return true; }
    n -= A * L;
    if (n < (N - L) * (A + B)) {
        s = n / (A + B); r = n % (A + B);
        if (r < B) return false;
        s += L; r -= B; return true;
    }
    n -= (N - L) * (A + B);
    s = N - L + n / B; r = n % B;
    return false;
}
constexpr u32 U_OUT = (128 - F) * 4, TAG_MASK = 0x3fffffu;
constexpr u32 U_INTRA = NSTEP * KEPT, U_CROSS_IN = NSTEP * (F + 8), U_CROSS = U_CROSS_IN + U_OUT;

template <int LMAX> __device__ __forceinline__ u32 vtime(u32 L, u32 s) {
    return s < EVEN_STEPS ? s : s + LMAX - L;
}
template <int LMAX, int L> __device__ __forceinline__ u32 before(u32 V) {
    return V <= EVEN_STEPS ? V : (u32)min(max((int)V - (LMAX - L), (int)EVEN_STEPS), NSTEP);
}
__device__ __forceinline__ void build_table(LAS u32* tab, u32 rank, u32 wid, u32 lane) {
    const u32 s = (wid & 1u) * 64u + lane, c = wid >> 1;
    {
        const u32 V = vtime<LEAD0>(c == 0u ? LEAD0 : c == 1u ? LEAD1 : c == 2u ? LEAD2 : 0u, s);
        const u32 pos = KC * (before<LEAD0, LEAD0>(V) + before<LEAD0, LEAD1>(V + (c < 1u)) + before<LEAD0, LEAD2>(V + (c < 2u)) +
                              before<LEAD0, 0>(V + (c < 3u)));
#pragma unroll
        for (u32 i = 0; i < KC; ++i) tab[pos + i] = s * 0x200001u + (c * KC + i) * 0x10080u;
    }
    {
        const bool w = c >= 2u;
        const u32 V = vtime<LEAD_W>(w ? LEAD_W : 0u, s);
        const u32 pos = 4096u + 8u * before<LEAD_W, LEAD_W>(V) + F * before<LEAD_W, 0>(V + w);
        const u32 n = w ? 8u : F, i0 = (c & 1u) ? n / 2u : 0u, i1 = (c & 1u) ? n : n / 2u;
        const u32 base = w ? ((Z_W + (rank & 3u) * 1024u) << 16) + V_W + s * 0x80008u : s * 0x200001u + KEPT * 0x10080u;
        for (u32 i = i0; i < i1; ++i) tab[pos + i] = base + i * (w ? 0x10001u : 0x10080u);
    }
#pragma unroll
    for (u32 n = wid * 64u + lane; n < 1024u; n += 512u)
        tab[4096u + U_CROSS_IN + n] = ((Z_O + (rank & 3u) * 1024u) << 16) | (V_O + n);
    if (wid < 2u) tab[U_INTRA + s] = 0u;
}
static_assert(2 * PER <= 128 && U_INTRA + 128 <= 4096 && U_CROSS_IN + 1024 <= 4096 && U_OUT + 2 * PER <= 1024, "table size");

__device__ __forceinline__ void pull_link(char* ws, char* out, const char* peer_pk, u32 dir, u32 p, u32 rank, u32 first,
                                          u32 hdr_par, u32 call, int lane, const LAS u32* tab, LAS u32* lock, LAS char* st) {
    const bool cross = dir >= 4;
    const u32 U = cross ? U_CROSS : U_INTRA, U0 = PUSH_STEPS * (cross ? 8 + F : KEPT);
    const auto ctrl = mkbuf(ws, CTRL_BYTES);
    const auto hdrs = mkbuf(ws + hdr_par + p * HDR_SRC, HDR_SRC);
    const auto flags = mkbuf(ws + OFF_FLAG + p * 32768, 32768);
    const auto src = mkbuf(peer_pk, PK_PAR);
    u32* const tmo = (u32*)(ws + OFF_TMO);
    tab += cross ? 4096 : 0;
    const u32 tick_off = lane == 0 ? (u32)OFF_TICKET + dir * 128 : 0xfffffff0u, lane0 = lane == 0 ? 0u : 0x7ffffff0u;
    const u32 tbase = (call - 1u) * (U - U0) - U0 - 2u * PER;
    const u32 dx = OFF_XRAW + p * XS, dw = OFF_WRAW + (p & 3) * WS - (Z_W + (rank & 3) * 1024) * 4096;
    const u32 vin = lane * 16, vout = out_lane(lane) + (p & 3) * 512;
    const u32 vlow = lane < 32 ? lane * 4 + 2560 : 0x7ffffff0u, vhigh = lane < 32 ? 0x7ffffff0u : lane * 4 - 128;
    const u64 side_off = (u64)(ws + OFF_SIDE + ((call & 1u) * 8 + p) * SIDE_SRC) - (u64)peer_pk;
    const u32 xtab = expand_table(lane);
    auto header = [&](u32 code) {
        const u32 v = code & 0xffffu;
        return __builtin_amdgcn_raw_buffer_load_b64(hdrs, v < V_O ? 0u : (lane & 15) * 8u, v < V_O ? v * 8 : HDR_OUT + (v - V_O) * 128, 17);
    };
    auto invalid = [&](u32x2 h) -> u32 {
        return (u32)__builtin_amdgcn_ballot_w64(((h[1] ^ call) & TAG_MASK) != 0u) & ((2u << inline_words(uni(h[0]))) - 1u);
    };
    u32 t = uni(first) + U0, c = uni(tab[t]);
    u32x2 g = header(c);
    bool gone = false;
    u32 pend = 8191 * 4;
    auto set_flag = [&]() { __builtin_amdgcn_raw_buffer_store_b32(call, flags, lane0, pend, 17); };
    asm volatile("" : "+v"(g));
    while (t < U) {
        u32 meta = uni(g[0]), tag = uni(g[1]), nm = 0u;
        if (invalid(g) != 0u) {
            WAIT_VM(0);
            set_flag();
            const u64 tw = __builtin_amdgcn_s_memrealtime();
            for (u32 it = 1;; ++it) {
                asm volatile("" ::: "memory");
                g = header(c);
                meta = uni(g[0]); tag = uni(g[1]);
                if (invalid(g) == 0u) break;
                if ((gone || (it & 63u) == 0u) && give_up(tmo, call, tw)) {
                    gone = true; meta = 0x80000000u; tag = rank << 29; nm = 0x7fc07fc0u;
                    break;
                }
                __builtin_amdgcn_s_sleep(1);
            }
        }
        const bool o1 = (c >> 16) >= Z_O, inl = o1 && inline_words(meta) != 0u, side = (meta & 0x2000u) != 0u;
        const u32 z = (c >> 16) + (o1 ? tag >> 22 : 0u), so = z * 4096;
        const u32 ntail = inl ? 0u : ((meta >> 15) & 0xffu) + ((meta >> 23) & 0x7fu);
        u32 hx = inl ? (u32)__builtin_amdgcn_update_dpp(0, (int)g[0], 0x101, 0xf, 0xf, true) : 0u;
        u32 vt = side ? vlow : lane * 4 + 2560;
        u32 vx = (side ? vhigh : lane + 64u < ntail ? lane * 4 + 2816 : 0x7ffffff0u) + (side ? (c & 0xfffcu) * 64 + (tag >> 22) * 4 : so);
        const auto xsrc = mkbuf(peer_pk + (side ? side_off : 0), side ? (u32)SIDE_SRC : (u32)PK_PAR);
        const auto dst = mkbuf(o1 ? out : ws, o1 ? (u32)(16 * MiB) : (u32)OFF_PK);
        u32 dv = o1 ? vout + out_unit(z & 1023u) : vin + (so + (z < Z_W ? dx : dw));
        auto dj = [&](int j) { return o1 ? out_piece(j) : j * 1024u; };
        asm volatile("" : "+v"(hx), "+v"(vt), "+v"(vx), "+v"(dv));
        auto take = [&]() {
#pragma nounroll
            for (u32 it = 0; it < 4096u; ++it) {
                u32 held = 1u;
                if (lane == 0) held = __hip_atomic_exchange(lock, 1u, __ATOMIC_RELAXED, __HIP_MEMORY_SCOPE_WORKGROUP);
                if (uni(held) == 0u) break;
                __builtin_amdgcn_s_sleep(2);
            }
            asm volatile("" ::: "memory");
        };
        const u32 flag = (c & 0xffffu) * 4;
        auto next = [&](u32 tk) {
            __hip_atomic_store(lock, 0u, __ATOMIC_RELAXED, __HIP_MEMORY_SCOPE_WORKGROUP);
            asm volatile("" : "+v"(tk) :: "memory");
            t = uni(tk) - tbase;
            c = uni(tab[t & 4095u]);
            g = header(c);
        };
        if (__builtin_expect((meta >> 31) == 0u, 1)) {
            take();
            u32x4 la = __builtin_amdgcn_raw_buffer_load_b128(src, vin, so, 17);
            u32x4 lb = __builtin_amdgcn_raw_buffer_load_b128(src, vin + 1024, so, 17);
            u32x2 lc = __builtin_amdgcn_raw_buffer_load_b64(src, lane * 8 + 2048, so, 17);
            u32 ta = __builtin_amdgcn_raw_buffer_load_b32(src, vt, so, 17);
            SCHED();
            u32 tx = __builtin_amdgcn_raw_buffer_load_b32(xsrc, vx, 0, 17);
            const u32 tk = __builtin_amdgcn_raw_ptr_buffer_atomic_add_i32(1, ctrl, tick_off, 0, 0);
            WAIT_VM(6);
            set_flag();
            asm volatile("" : "+v"(la), "+v"(lb), "+v"(lc), "+v"(ta), "+v"(tx) :: "memory");
            next(tk);
            u32x4 o[4];
            unpack_unit(o, la, lb, lc, side ? ta | tx : ta, tx | hx, meta, xtab, st, lane);
            asm volatile("" : "+v"(g));
#pragma unroll
            for (int j = 0; j < 4; ++j) __builtin_amdgcn_raw_buffer_store_b128(o[j], dst, dv + dj(j), 0, 17);
            asm volatile("; packed" ::: "memory");
        } else {
            u32x4 o[4];
            take();
#pragma unroll
            for (int j = 0; j < 4; ++j) o[j] = __builtin_amdgcn_raw_buffer_load_b128(src, vin + j * 1024, so, 17);
            const u32 tk = __builtin_amdgcn_raw_ptr_buffer_atomic_add_i32(1, ctrl, tick_off, 0, 0);
            WAIT_VM(5);
            set_flag();
            asm volatile("" : "+v"(o[0]), "+v"(o[1]), "+v"(o[2]) :: "memory");
            next(tk);
            asm volatile("" : "+v"(g));
#pragma unroll
            for (int j = 0; j < 4; ++j) __builtin_amdgcn_raw_buffer_store_b128(o[j] | nm, dst, dv + dj(j), 0, 17);
        }
        asm volatile("" ::: "memory");
        pend = flag;
    }
    WAIT_VM(0);
    set_flag();
}

__device__ __forceinline__ void glds16(const char* sbase, u32 voff, u32 lds_addr) {
    asm volatile("s_mov_b32 m0, %2\n\ts_nop 0\n\tglobal_load_lds_dwordx4 %0, %1 sc0 sc1"
                 :: "v"(voff), "s"(sbase), "s"(lds_addr) : "memory");
}
__device__ __forceinline__ void glds16_half(const char* sbase, u32 voff, u32 lds_addr) {
    asm volatile("s_mov_b32 m0, %2\n\ts_mov_b32 exec_hi, 0\n\tglobal_load_lds_dwordx4 %0, %1 sc0 sc1\n\ts_mov_b32 exec_hi, -1"
                 :: "v"(voff), "s"(sbase), "s"(lds_addr) : "memory");
}

__device__ __forceinline__ u32 silu2(float y0, float y1) {
    f32x2 c = {__builtin_amdgcn_fmed3f(y0, -60.f, 60.f), __builtin_amdgcn_fmed3f(y1, -60.f, 60.f)};
    c *= -1.44269504088896f;
    f32x2 d = {__builtin_amdgcn_exp2f(c[0]), __builtin_amdgcn_exp2f(c[1])};
    d += 1.f;
    f32x2 r = {__builtin_amdgcn_rcpf(d[0]), __builtin_amdgcn_rcpf(d[1])};
    r *= (f32x2){y0, y1};
    return __builtin_bit_cast(u32, __builtin_convertvector(r, bf16x2));
}

__global__ void __launch_bounds__(576) ag_gemm_silu(const Params P) {
    __shared__ __attribute__((aligned(16))) char lds_raw[LDS_BYTES];
    LAS char* lds = (LAS char*)lds_raw;
    const int lane = threadIdx.x & 63, wid = uni(threadIdx.x >> 6);
    int rank_ = P.rank;
    u64 ws_ = (u64)P.self, xin_ = (u64)P.x, win_ = (u64)P.w, tile_ = P.tile[blockIdx.x];
    asm volatile("" : "+s"(rank_), "+s"(ws_), "+s"(xin_), "+s"(win_), "+s"(tile_));
    typedef __attribute__((address_space(1))) char gchar;
    const int rank = rank_, oth = (rank & 4) ^ 4;
    char* const ws = (char*)(gchar*)ws_;
    const char* const xin = (const char*)(gchar*)xin_;
    const u64 tile = tile_, w_off = win_ - xin_;
    const u32 cb = (u32)tile & 0xffu, ridx = (u32)(tile >> 56);
    u32* const tmo = (u32*)(ws + OFF_TMO);
    u32* const ctr = (u32*)(ws + OFF_CTR) + blockIdx.x * 16 + wid;
    auto call_number = [&]() { u64 a = (u64)ctr; asm volatile("" : "+s"(a)); return *(const __attribute__((address_space(4))) u32*)a + 1u; };
    u32 job = NJOBS, ja = 0, jb = 0, jsrc = 0;
    bool jx = false;
    u32x4 nx[4];
    char* jpeer;
    auto job_load = [&]() {
        const auto in = mkbuf(xin + (jx ? 0 : w_off), 16 * MiB);
#pragma unroll
        for (int j = 0; j < 4; ++j) nx[j] = __builtin_amdgcn_raw_buffer_load_b128(in, jsrc, j * (jx ? 131072 : 4096), 0);
        jsrc += jx ? 128 : 16384;
    };
    auto job_start = [&]() {
        jsrc = (jx ? (lane >> 3) * 16384 + (lane & 7) * 16 : (lane >> 5) * 2048 + (lane & 31) * 16) + (jx ? ja * 524288 + jb * 128 : jb * 16384 + ja * 512);
        job_load();
        const u32 q = (lane >> 2) & 3;
        jpeer = P.ws[(!jx ? oth | ja : ja < KEPT ? rank ^ (q + 1) : oth | q) & 7];
    };
    auto group_start = [&]() {
        constexpr u32 KE = 3 * KC;
        u32 c, r;
        const bool a = order<KE + 32, 32 - KE, 5, NCHUNK - HEAD_STEPS / 4>(job - HEAD_JOBS, c, r);
        c += HEAD_STEPS / 4;
        jx = !a || r < KE;
        ja = !a ? KE + r : r < KE ? r : (r - KE) & 3;
        jb = jx ? c * 4 : (c * 8 + ((r - KE) >> 2)) * 4;
        job_start();
    };
    constexpr u32 PW = KEPT > F + 8 ? KEPT : F + 8;
    const u32 pj = wid * NLOC + ridx, pdir = pj % 7 + 1, pi = pj / 7 % PW, ps = pj / (7 * PW);
    bool push = false;
    {
        const u32 j = PUSH_STEPS * 64 + wid * NREM + ridx;
        if (cb - 4u < 4u && wid < 8 && j < HEAD_JOBS) {
            job = j; jx = (j & 32u) == 0u; ja = j & (jx ? 31u : 3u); jb = jx ? j >> 6 : (j >> 6) * 8 + ((j >> 2) & 7u);
        } else if (cb < 4 && pj < PUSH_STEPS * 7 * PW && pi < (pdir < 4 ? KEPT : F + 8)) {
            push = true; jx = pdir < 4 || pi < F; ja = pdir < 4 ? pi : jx ? KEPT + pi : (rank ^ pdir) & 3; jb = jx ? ps : ps * 8 + pi - F;
        }
        if (job < NJOBS || push) job_start();
    }
    const u32 call = call_number(), par = call & 1u;
    if (cb < 4 && wid < 8) {
        if (push) {
            const u32 dst = jx ? OFF_XRAW + rank * XS + ((ps + par * 128) * 32 + ja) * 4096 : OFF_WRAW + (rank & 3) * WS + (jb + par * 1024) * 4096;
            const u32 flag = OFF_FLAG + (rank * 8192 + (jx ? ja * 128 + ps : V_W + jb)) * 4;
            const auto peer = mkbuf(P.ws[(rank ^ pdir) & 7], OFF_PK);
#pragma unroll
            for (int j = 0; j < 4; ++j) __builtin_amdgcn_raw_buffer_store_b128(nx[j], peer, dst + j * 1024 + lane * 16, 0, 17);
            WAIT_VM(0);
            __builtin_amdgcn_raw_buffer_store_b32(call, peer, lane == 0 ? flag : 0xfffffff0u, 0, 17);
        }
        job = HEAD_JOBS + (7 - wid) * NLOC + ridx;
        if (wid >= 4) __builtin_amdgcn_s_setprio(1);
        for (u32 i = 0; i < 2 + (job - HEAD_JOBS) / 60; ++i) __builtin_amdgcn_s_sleep(8);
        group_start();
    }
    if (lane == 0) __hip_atomic_store(ctr, call, __ATOMIC_RELAXED, __HIP_MEMORY_SCOPE_AGENT);
    const size_t hdr_par = OFF_HDR + par * HDR_PAR;

    if (cb == 0xffu) {
        if (wid < 8) build_table((LAS u32*)(lds + LDS_TABLE), rank, wid, lane);
        else ((LAS u32*)(lds + LDS_LOCK))[lane & 3] = 0u;
        WAIT_LGKM0();
        __builtin_amdgcn_s_barrier();
        asm volatile("" ::: "memory");
        if (wid == 8 && HEAD_JOBS + NLOC * (8 + ROUND2) + NLATE * 8 + ridx < NJOBS) {
            job = HEAD_JOBS + NLOC * (8 + ROUND2) + NLATE * 8 + ridx;
            for (int i = 0; i < 3; ++i) __builtin_amdgcn_s_sleep(127);
            group_start();
        }
    }

    const auto pk = mkbuf(ws + OFF_PK + par * PK_PAR, PK_PAR);
    const u32 ctab = compact_table(lane);
    LAS char* const st = lds + LDS_CODEC + wid * CODEC_BYTES;
    if (job < HEAD_JOBS) {
        u32 t0, t1, nt;
        const u32 meta = pack_unit(nx, choose_window(nx[0]), false, ctab, st, pk, (jx ? jb * 32 + ja : Z_W + ja * 1024 + jb) * 4096, lane, t0, t1, nt, []() {});
        u64* const hp = (u64*)(jpeer + hdr_par + rank * HDR_SRC) + (jx ? ja * 128 + jb : V_W + jb);
        WAIT_VM(0);
        if ((lane & 3) == 0 && (u32)(lane >> 2) < (!jx ? 1u : ja < KEPT ? 3u : 4u))
            __hip_atomic_store(hp, ((u64)(call & TAG_MASK) << 32) | meta, __ATOMIC_RELAXED, __HIP_MEMORY_SCOPE_SYSTEM);
        job = NJOBS;
    }
    if (cb - 4u < 4u && wid < 8 && ridx < NLATE) {
        job = HEAD_JOBS + NLOC * (8 + ROUND2) + wid * NLATE + ridx;
        if (job < NJOBS) group_start();
    }
    while (job < NJOBS) {
        constexpr u32 jn = 4;
        const u32 slot = jx ? jb * 32 + ja : Z_W + ja * 1024 + jb;
        const u32 win = choose_window(nx[0]);
        u32 hd = 0, ht = call & TAG_MASK, sw = 0, ns_ = 0;
#pragma nounroll
        for (u32 u = 0; u < jn; ++u) {
            const u32x4 r[4] = {nx[0], nx[1], nx[2], nx[3]};
            if (u + 1 < jn) job_load();
            u32 t0, t1, nt;
            u32 meta = pack_unit(r, win, true, ctab, st, pk, (slot + u * (jx ? 32 : 1)) * 4096, lane, t0, t1, nt,
                                 [&]() { asm volatile("" : "+v"(nx[0]), "+v"(nx[1]), "+v"(nx[2]), "+v"(nx[3])); });
            const u32 more = nt > 32u ? nt - 32u : 0u;
            if ((meta & 0x1000u) && ns_ + more <= 64u) {
                const u32 w = (u32)__builtin_amdgcn_ds_bpermute((int)((lane + 32u - ns_) * 4u), (int)t0);
                if ((u32)lane >= ns_) sw = w;
                meta |= 0x2000u;
                if ((u32)(lane & 3) == u) ht |= ns_ << 22;
                ns_ += more;
            }
            if ((u32)(lane & 3) == u) hd = meta;
        }
        const u32 ncons = !jx ? 1u : ja < KEPT ? 3u : 4u, v0 = jx ? ja * 128 + jb : V_W + jb;
        if (ns_ != 0u) {
            u32x4 q;
#pragma unroll
            for (int k = 0; k < 4; ++k) q[k] = (u32)__builtin_amdgcn_ds_bpermute((lane & 15) * 16 + k * 4, (int)sw);
            const u64 pb = (u64)jpeer;
            const u64 pp = ((u64)(u32)__builtin_amdgcn_ds_bpermute(lane & 48, (int)(pb >> 32)) << 32 | (u32)__builtin_amdgcn_ds_bpermute(lane & 48, (int)pb)) +
                           OFF_SIDE + (par * 8 + rank) * SIDE_SRC + v0 * 64 + (lane & 15) * 16;
            if ((u32)(lane >> 4) < ncons && (u32)(lane & 14) * 4 < ns_)
                asm volatile("global_store_dwordx4 %0, %1, off sc0 sc1\n\ts_nop 1" :: "v"(pp), "v"(q) : "memory");
        }
        const bool on = (u32)(lane & 3) < jn && (u32)(lane >> 2) < ncons;
        u64* const hp = (u64*)(jpeer + hdr_par + rank * HDR_SRC) + v0 + (lane & 3);
        job = wid == 8 ? job + NCOMM : cb < 4 && job < HEAD_JOBS + NLOC * ROUND2 ? job + NLOC * 8 : NJOBS;
        WAIT_VM(0);
        if (on) __hip_atomic_store(hp, ((u64)ht << 32) | hd, __ATOMIC_RELAXED, __HIP_MEMORY_SCOPE_SYSTEM);
        __builtin_amdgcn_s_setprio(0);
        if (job < NJOBS) group_start();
    }
    if (cb == 0xffu) {
        const u32 gw = (wid & 3) * NCOMM + ridx;
        if (wid < 8 && gw < PER * 7) {
            const u32 dir = gw % 7 + 1, p = (rank ^ dir) & 7u;
            pull_link(ws, P.out, P.ws[p] + OFF_PK + par * PK_PAR, dir, p, rank, gw / 7 + (wid >> 2) * PER, (u32)hdr_par, call, lane,
                      (const LAS u32*)(lds + LDS_TABLE), (LAS u32*)(lds + LDS_LOCK) + (wid & 3), lds + wid * UNPACK_BYTES);
        }
        return;
    }

    const int wm = wid >> 2, wn = wid & 3, fr = lane & 15, fq = lane >> 4;
    const u32 ns = (u32)(tile >> 8) & 0xffu;
    auto strip_of = [&](u32 i) -> u32 { return (u32)(tile >> (16 + 8 * i)) & 0xffu; };
    if (wid == 8) {
        const u32 i = lane & 15, jj = lane >> 4;
        const u32 gs = strip_of(i < 5 ? i : 0);
        const u32 off = OFF_FLAG + (i < 5 ? (gs >> 5) * 8192 + (gs & 31) * 128 + jj : (oth | (cb & 3)) * 8192 + V_W + jj * 8 + ((i - 5) & 7)) * 4;
        const u32 sh = i < 5 ? 2 : 5;
        const u64 skip = __builtin_amdgcn_ballot_w64(i < 5 ? (gs >> 5) == (u32)rank : (cb < 4 || i >= 13));
        const auto ctrl = mkbuf(ws, CTRL_BYTES);
        const u64 t_in = __builtin_amdgcn_s_memrealtime();
        u32 ready = skip == ~0ull ? NSTEP : 0;
        for (u32 s = 0; s < NSTEP; ++s) {
            if (cb < 4) while (__builtin_amdgcn_s_memrealtime() - t_in < PACE_T0 + s * PACE_STEP) __builtin_amdgcn_s_sleep(2);
            if (s >= ready) {
                const u64 t0 = __builtin_amdgcn_s_memrealtime();
                for (u32 it = 1;; ++it) {
                    asm volatile("" ::: "memory");
                    const u32 v = __builtin_amdgcn_raw_buffer_load_b32(ctrl, off + (s << sh), 0, 17);
                    const u64 m = ~(__builtin_amdgcn_ballot_w64((int)(v - call) >= 0) | skip);
                    ready = s + (m == 0ull ? 4u : (u32)__builtin_ctzll(m) >> 4);
                    if (s < ready) break;
                    if ((it & 63u) == 0u && give_up(tmo, call, t0)) { WAIT_VM(0); ready = NSTEP; break; }
                    __builtin_amdgcn_s_sleep(2);
                }
            }
            asm volatile("" ::: "memory");
            __builtin_amdgcn_s_barrier();
            if (s == 2) __builtin_amdgcn_s_barrier();
        }
        return;
    }
    const u32 s01 = strip_of((wid & 2) + wm), h01 = wid & 1, s2 = strip_of(4);
    const bool own01 = (s01 >> 5) == (u32)rank, own2 = (s2 >> 5) == (u32)rank;
    const char* gA01 = own01 ? xin + (size_t)(s01 & 31) * (32 * 16384) : ws + OFF_XRAW + (s01 >> 5) * XS + (s01 & 31) * 4096;
    const char* gA2 = own2 ? xin + (size_t)(s2 & 31) * (32 * 16384) : ws + OFF_XRAW + (s2 >> 5) * XS + (s2 & 31) * 4096;
    const char* gB = cb < 4 ? xin + w_off + cb * 512 : ws + OFF_WRAW + (cb & 3) * WS;
    const u32 dA01 = own01 ? 128u : 131072u, dA2 = own2 ? 128u : 131072u, dB = cb < 4 ? 131072u : 32768u;
    const u32 rs01 = own01 ? 16384u : 128u, rs2 = own2 ? 16384u : 128u, rsB = cb < 4 ? 2048u : 512u;
    const u32 l3 = lane >> 3;
    const u32 vA0 = (h01 * 16 + l3) * rs01 + (((lane & 7) ^ (l3 >> 1)) * 16);
    const u32 vA1 = (h01 * 16 + 8 + l3) * rs01 + (((lane & 7) ^ (4 + (l3 >> 1))) * 16);
    const u32 vA2 = (4 * wid + (l3 & 3)) * rs2 + (((lane & 7) ^ (((4 * wid + (l3 & 3)) >> 1) & 7)) * 16);
    u32 vB[2];
#pragma unroll
    for (int i = 0; i < 2; ++i) {
        const u32 k = 8 * wid + 2 * i + (lane >> 5), f = (k & 3) | (((k >> 3) & 1) << 2);
        vB[i] = k * rsB + (((lane & 31) ^ (f << 1)) * 16);
    }
    const u32 lds0 = (u32)(size_t)lds;
    const u32 rA0 = (wm * 80 + fr) * 128 + ((fq ^ (fr >> 1)) * 16), rA1 = rA0 ^ 64;
    u32 rB[4];
    {
        const u32 q = fr >> 2, p4 = fr & 3, f = q | ((fq & 1) << 2);
#pragma unroll
        for (int n = 0; n < 4; ++n) rB[n] = A_BYTES + (8 * fq + q) * 512 + (((wn * 4 + n) ^ f) * 32) + p4 * 8;
    }

    auto stage_piece = [&](int i, u32 slot_base) {
        if (i < 2) glds16(gA01, i ? vA1 : vA0, slot_base + (wid + wm) * 2048 + i * 1024);
        else if (i < 6) glds16(gB + (i >= 4 ? 4 * rsB : 0u), vB[i & 1], slot_base + A_BYTES + wid * 4096 + (i - 2) * 1024);
        else glds16_half(gA2, vA2, slot_base + 8192 + wm * 8192 + wid * 512);
    };
    auto advance = [&]() { gA01 += dA01; gA2 += dA2; gB += dB; };
    auto shift = [&](int sign) {
        const long long o = sign * (long long)(par * 128);
        gA01 += own01 ? 0 : o * 131072; gA2 += own2 ? 0 : o * 131072; gB += cb < 4 ? 0 : o * 32768;
    };

    f32x4 acc[5][4];
#pragma unroll
    for (int m = 0; m < 5; ++m)
#pragma unroll
        for (int n = 0; n < 4; ++n) acc[m][n] = (f32x4){0.f, 0.f, 0.f, 0.f};
    s16x8 fa[2][5];
    s16x4 fb[2][4][2];
    auto frag_read = [&](int set, int i, u32 sb, int kk) {
        if (i == 2) fa[set][0] = *(const LAS s16x8*)(lds + sb + (kk ? rA1 : rA0));
        else if (i >= 9) fa[set][i - 8] = *(const LAS s16x8*)(lds + sb + (kk ? rA1 : rA0) + (i - 8) * 2048);
        else {
            const int b = i < 2 ? i : i - 1, n = b >> 1, t = b & 1;
            fb[set][n][t] = __builtin_amdgcn_ds_read_tr16_b64_v4i16((LAS s16x4*)(lds + sb + rB[n] + t * 2048 + kk * 16384));
        }
    };
    auto mma = [&](int set, int i) {
        const int m = i >> 2, n = i & 3;
        const s16x8 b = __builtin_shufflevector(fb[set][n][0], fb[set][n][1], 0, 1, 2, 3, 4, 5, 6, 7);
        acc[m][n] = __builtin_amdgcn_mfma_f32_16x16x32_bf16(__builtin_bit_cast(bf16x8, b), __builtin_bit_cast(bf16x8, fa[set][m]), acc[m][n], 0, 0, 0);
    };

    u32 cur = 0, tmo_v;
#define KSTEP(MB, LOAD, VMW) do { \
        constexpr int H = 4 * MB, NR = 8 + MB, BP = MB == 5 ? 4 : 0, GS = MB == 5 ? 20 : 3; \
        const u32 nxt = cur == 2 * STAGE ? 0u : cur + STAGE; \
        int nr = 0, ng = 0; \
        _Pragma("unroll") for (int i = 0; i < 2 * H; ++i) { \
            if (i == H - BP) { if (LOAD || t == 0) WAIT_VM(VMW); else WAIT_VM(0); WAIT_LGKM0(); __builtin_amdgcn_s_barrier(); asm volatile("" ::: "memory"); SCHED(); } \
            mma(i >= H, i % H); SCHED(); \
            if (i < H) { \
                if (nr < NR) { frag_read(1, nr, cur, 1); ++nr; } \
                if (i >= H - BP && (i & 1)) { if (LOAD) stage_piece(ng, lds0 + cur); ++ng; } \
                if (i == H - 1) nr = 0; \
            } else { \
                const bool g = ng < MB + 2 && ((i - H) % GS == GS - 1 || nr >= NR || 2 * H - i <= MB + 2 - ng); \
                if (nr < NR && (!g || 2 * H - i <= NR - nr + 1)) { frag_read(0, nr, nxt, 0); ++nr; } \
                if (g) { if (LOAD) stage_piece(ng, lds0 + cur); ++ng; } \
            } \
            SCHED(); } \
        if (LOAD) advance(); \
        cur = nxt; } while (0)
#define KLOOP(MB, NG, NG2) do { \
        __builtin_amdgcn_s_waitcnt(0x0F70);     \
        _Pragma("unroll") for (int s = 0; s < 3; ++s) { \
            __builtin_amdgcn_s_barrier(); \
            if (s == 0 && PUSH_STEPS > 0) shift(1); \
            _Pragma("unroll") for (int i = 0; i < NG; ++i) stage_piece(i, lds0 + s * STAGE); \
            advance(); \
            if (s + 1 == PUSH_STEPS) shift(-1); } \
        WAIT_VM(NG2); \
        __builtin_amdgcn_s_barrier(); \
        asm volatile("" ::: "memory"); \
        _Pragma("unroll") for (int i = 0; i < 8 + MB; ++i) frag_read(0, i, 0, 0); \
        for (u32 t = 0; t < NSTEP - 3; ++t) KSTEP(MB, true, NG); \
        tmo_v = __hip_atomic_load(tmo, __ATOMIC_RELAXED, __HIP_MEMORY_SCOPE_AGENT);     \
        _Pragma("nounroll") for (u32 t = 0; t < 3; ++t) KSTEP(MB, false, NG); } while (0)
    if (ns > 4) KLOOP(5, 7, 14); else KLOOP(4, 6, 12);
#undef KLOOP
#undef KSTEP

    const u32 nanm = tmo_v == call ? 0x7fc07fc0u : 0u;
    int tz = threadIdx.x;
    asm volatile("" : "+v"(tz));
    const int ln = tz & 63;
    auto act = [&](int m, u32x4& lo, u32x4& hi) {
        u32 o[4][2];
#pragma unroll
        for (int n = 0; n < 4; ++n) { o[n][0] = silu2(acc[m][n][0], acc[m][n][1]) | nanm; o[n][1] = silu2(acc[m][n][2], acc[m][n][3]) | nanm; }
        const auto a0 = __builtin_amdgcn_permlane16_swap(o[0][0], o[1][0], false, false), a1 = __builtin_amdgcn_permlane16_swap(o[0][1], o[1][1], false, false);
        const auto b0 = __builtin_amdgcn_permlane16_swap(o[2][0], o[3][0], false, false), b1 = __builtin_amdgcn_permlane16_swap(o[2][1], o[3][1], false, false);
        lo = (u32x4){a0[0], a1[0], a0[1], a1[1]};
        hi = (u32x4){b0[0], b1[0], b0[1], b1[1]};
    };
    const auto out = mkbuf(P.out, 16 * MiB);
    const u32 q4 = cb & 3, ctab2 = compact_table(ln);
    u64* const hbase = (u64*)(P.ws[(oth | q4) & 7] + hdr_par + rank * HDR_SRC + HDR_OUT);
    LAS u32x4* const half = (LAS u32x4*)(lds + wn * 2048) + ln;
    u32 win = 0;
    u32x4 r[4], nr[2];
    asm volatile("" ::: "memory");
    if (wm == 1) __builtin_amdgcn_s_setprio(1);
    act(0, nr[0], nr[1]);
    auto phase = [&](auto PH) {
        constexpr u32 ph = decltype(PH)::value;
        r[0] = nr[0]; r[1] = nr[1];
        if (ph < 2) act(2 * ph + 1, r[2], r[3]);
        else { __builtin_amdgcn_s_barrier(); asm volatile("" ::: "memory"); r[2] = half[0]; r[3] = half[64]; }
        const u32 pos = ph < 2 ? 2 * ph + wm : 4, e = strip_of(pos) * 4 + wn;
        const bool on = pos < ns;
        u32 meta = 0, over = 0;
        if (on) {
            if (cb < 4) {
#pragma unroll
                for (int j = 0; j < 4; ++j)
                    __builtin_amdgcn_raw_buffer_store_b128(r[j], out, out_lane(ln) + out_unit(e) + cb * 512 + out_piece(j), 0, 16);
            } else {
                if (ph == 0) win = choose_window(r[0]);
                u32 t0, nt;
                meta = pack_unit(r, win, false, ctab2, st, pk, (Z_O + q4 * 1024 + e) * 4096, ln, t0, over, nt, []() {});
            }
        }
        asm volatile("" ::: "memory");
        u32 seq = __builtin_amdgcn_raw_ptr_buffer_atomic_add_i32(1, mkbuf(ws, CTRL_BYTES), ln == 0 && on && cb >= 4 ? (u32)OFF_OSEQ + q4 * 128 : 0xfffffff0u, 0, 0);
        asm volatile("" ::: "memory");
        SCHED();
        if (ph == 0) act(2, nr[0], nr[1]);
        if (ph == 1 && ns > 4) {
            act(4, nr[0], nr[1]);
            if (wm == 1) { half[0] = nr[0]; half[64] = nr[1]; }
        }
        if (ph < 2) asm volatile("" : "+v"(nr[0]), "+v"(nr[1]));
        WAIT_VM(0);
        asm volatile("" : "+v"(seq));
        if (on && cb >= 4) {
            const u32 w = (u32)__builtin_amdgcn_update_dpp((int)meta, (int)over, 0x111, 0xf, 0xf, false);
            if ((u32)ln <= (inline_words(meta) | 3u))
                __hip_atomic_store(hbase + ((uni(seq) - (call - 1u) * U_OUT) & 1023u) * 16 + ln, ((u64)((call & TAG_MASK) | (e << 22)) << 32) | w,
                                   __ATOMIC_RELAXED, __HIP_MEMORY_SCOPE_SYSTEM);
        }
        SCHED();
    };
    phase(Const<0>());
    phase(Const<1>());
    if (ns > 4) {
        if (wm == 0) phase(Const<2>());
        else { WAIT_LGKM0(); __builtin_amdgcn_s_barrier(); }
    }
}

static bool build_tiles(int rank, u64* tile) {
    const int oth = (rank & 4) ^ 4;
    u64 rows[8][40];
    int key[8][40], nrow[8];
    for (int cb = 0; cb < 8; ++cb) {
        int q[160], pos[160], nq = 0;
        auto add = [&](int gs, int p) { q[nq] = gs; pos[nq++] = p; };
        if (cb < 4) for (int t = KEPT; t < 32; ++t) add(rank * 32 + t, 0);
        for (int t = 0; t < KEPT; ++t) add(rank * 32 + t, 20 + t);
        for (int c = 0; c < 3; ++c) for (int d = 1; d < 4; ++d) for (int t = c * KC; t < c * KC + KC; ++t) add((rank ^ d) * 32 + t, 100 + c * 200 + d * 32 + t);
        for (int i = 0; i < 4; ++i) if (cb < 4 || (cb & 3) != i) for (int t = KEPT; t < 32; ++t) add((oth | i) * 32 + t, 700 + i * 32 + t);
        for (int d = 1; d < 4; ++d) for (int t = 3 * KC; t < KEPT; ++t) add((rank ^ d) * 32 + t, 900 + d * 32 + t);
        nrow[cb] = 0;
        while (nq > 0) {
            int left = 0;
            auto grp = [&](int i) { return cb < 4 ? 0 : pos[i] < 700 ? 0 : pos[i] < 900 ? 1 : 2; };
            while (left < nq && grp(nq - 1 - left) == grp(nq - 1)) ++left;
            const int nt = (left + 4) / 5, n = (left + nt - 1) / nt, s0 = nq - n;
            u64 e = (u64)cb | ((u64)n << 8);
            for (int s = 0; s < 5; ++s) e |= (u64)q[nq - 1 - (s < n ? s : 0)] << (16 + 8 * s);
            key[cb][nrow[cb]] = pos[s0 + n / 2];
            rows[cb][nrow[cb]++] = e;
            nq = s0;
        }
    }
    u64 list[8][32];
    int nl[8];
    for (int half = 0; half < 2; ++half) {
        int total = 0, n = 0;
        for (int c = 0; c < 4; ++c) { total += nrow[half * 4 + c]; nl[half * 4 + c] = 0; }
        for (int k = 0; k < 1100; ++k)
            for (int c = 0; c < 4; ++c) {
                const int cb = half * 4 + c;
                for (int r = 0; r < nrow[cb]; ++r)
                    if (key[cb][r] == k) { const int x = half * 4 + n * 4 / total; list[x][nl[x]++ & 31] = rows[cb][r]; ++n; }
            }
    }
    int ng = 0, nc = 0;
    for (int b = 0; b < NWG; ++b) {
        const int xcd = b & 7, i = (b >> 3) - (32 - nl[xcd]);
        tile[b] = i < 0 ? 0xffull | ((u64)nc++ << 56) : list[xcd][i];
    }
    int nr = 0;
    for (int cls = 4; cls >= 0; --cls)
        for (int b = 0; b < NWG; ++b) {
            const int cb = (int)(tile[b] & 0xff), gs = (int)((tile[b] >> 16) & 0xff), t = gs & 31;
            if (cb == 0xff) continue;
            if (cb < 4) { if (cls == 0) tile[b] |= (u64)ng++ << 56; }
            else if (cls == ((gs >> 5) == rank ? 0 : t >= 3 * KC ? 4 : 1 + t / KC)) tile[b] |= (u64)nr++ << 56;
        }
    return ng == NLOC && nr == NREM && nc == NCOMM;
}
}

extern "C" void kernel_launch(int rank, int nranks, ncclComm_t comm,
                              hipStream_t stream,
                              void* const* d_in, const int* in_sizes, int n_in,
                              void* d_out, int out_size,
                              const int* in_dtypes, int out_dtype,
                              void* const* d_out_all,
                              void* const* d_ws_all, size_t ws_bytes) {
    using namespace agk;
    if (nranks != NRANK || (unsigned)rank >= (unsigned)NRANK || ws_bytes < WS_NEED || n_in < 2 ||
        in_sizes[0] != 1024 * KDIM || in_sizes[1] != KDIM * 1024 || out_size != 8192 * 1024) {
        fprintf(stderr, "kernel_launch: unexpected problem shape\n");
        return;
    }
    Params P;
    for (int p = 0; p < NRANK; ++p) P.ws[p] = (char*)d_ws_all[p];
    P.x = (const char*)d_in[0];
    P.w = (const char*)d_in[1];
    P.out = (char*)d_out;
    P.rank = rank;
    P.self = P.ws[rank];
    P.pad = 0;
    if (!build_tiles(rank, P.tile)) { fprintf(stderr, "kernel_launch: tile table does not match the role counts\n"); return; }
    hipLaunchKernelGGL(ag_gemm_silu, dim3(NWG), dim3(576), 0, stream, P);
    HIP_CHECK(hipGetLastError());
}
```
